```python
import numpy as np

kernel_name = "gpu_ag_gemm_m4096_k4096_n32768_f32_gelu"


ABSMAX_RTOL = 0.001

COMPUTE_DTYPE = "float32"
EPILOGUE = "gelu"
IN_QUANT = None
OUT_QUANT = None
IN_DTYPES = {"x": "float32", "w_mat": "float32"}
WORLD_SIZE = 8

def setup_inputs():

    x = np.random.randn(4096, 4096).astype(np.float32)
    w_mat = (np.random.randn(4096, 32768) / np.sqrt(4096)).astype(np.float32)
    return {"x": x, "w_mat": w_mat}

def _epilogue(y):
    c = 0.7978845608028654
    return (0.5 * y * (1.0 + np.tanh(c * (y + 0.044715 * y ** 3)))).astype(np.float32)

def reference(x, w_mat):
    y = (x @ w_mat).astype(np.float32)
    return _epilogue(y)
```

```cpp
#include <hip/hip_runtime.h>
#include <rccl/rccl.h>
#include <cstdio>

#define WNT " nt"
#define UNL(x) __builtin_expect(!!(x), 0)
#define LAS __attribute__((address_space(3)))
#define GAS __attribute__((address_space(1)))
typedef _Float16 half8 __attribute__((ext_vector_type(8)));
typedef float f32x4 __attribute__((ext_vector_type(4)));
typedef unsigned u32x4 __attribute__((ext_vector_type(4)));
typedef float f32x2 __attribute__((ext_vector_type(2)));
#define NO_EARLY
#define PF_NONE
#define SHIFT 0

constexpr int NRANK = 8;
constexpr int M_ = 4096, K_ = 4096, N_ = 4096, MS = M_ / NRANK;
constexpr int NT = K_ / 64;
constexpr int HTB = 16384;
constexpr int LDS_BYTES = 8 * HTB + 512;
constexpr size_t OFF_A16 = 1u << 20;
constexpr size_t A16_BYTES = (size_t)MS * K_ * 2;
constexpr size_t OFF_B16 = 16u << 20;
constexpr size_t WS_NEED = OFF_B16 + (size_t)K_ * N_ * 2 + (1u << 20);
constexpr unsigned long long DEADLINE = 50000000ull;
constexpr unsigned CB_TMO = 0;
constexpr unsigned CB_TK = 4096;
constexpr unsigned CB_BC = CB_TK + 256 * 128;
constexpr unsigned CB_AC = CB_BC + 512 * 128;

#define STAMPW(i, w) do {} while (0)
#define STAMP(i) STAMPW(i, 0)
#define OAUX 2
#define OSTORE(v, ptr) __builtin_nontemporal_store(v, ptr)
struct Params { const float* x; const float* w; float* out; char* ws[NRANK]; int rank; int phase; };

__device__ __forceinline__ half8 cvt8(f32x4 a, f32x4 b) {
    half8 h; h[0] = (_Float16)a[0]; h[1] = (_Float16)a[1]; h[2] = (_Float16)a[2]; h[3] = (_Float16)a[3];
    h[4] = (_Float16)b[0]; h[5] = (_Float16)b[1]; h[6] = (_Float16)b[2]; h[7] = (_Float16)b[3]; return h;
}
__device__ __forceinline__ float gelu1(float y) {
    const float k1 = -2.0f * 0.7978845608028654f * 1.4426950408889634f, k2 = k1 * 0.044715f;
    float z = y * __builtin_fmaf(y * y, k2, k1);
    float e = __builtin_amdgcn_exp2f(z);
    return y * __builtin_amdgcn_rcpf(1.0f + e);
}
__device__ __forceinline__ bool wait_words(const GAS unsigned* word, unsigned need, unsigned* tmo, unsigned call) {
    const unsigned long long t0 = __builtin_amdgcn_s_memrealtime();
    for (unsigned n = 1;; ++n) {
        const unsigned v = __hip_atomic_load(word, __ATOMIC_RELAXED, __HIP_MEMORY_SCOPE_SYSTEM);
        if (__all((int)(v - need) >= 0)) return false;
        if ((n & 15u) == 0u) {
            if (__builtin_amdgcn_readfirstlane(__hip_atomic_load(tmo, __ATOMIC_RELAXED, __HIP_MEMORY_SCOPE_AGENT)) == call || __builtin_amdgcn_s_memrealtime() - t0 > DEADLINE) {
                __hip_atomic_store(tmo, call, __ATOMIC_RELAXED, __HIP_MEMORY_SCOPE_AGENT); return true; }
        }
        __builtin_amdgcn_s_sleep(1);
    }
}

__global__ __launch_bounds__(512, 2) __attribute__((amdgpu_num_vgpr(220))) void ag_gemm(Params p) {
    extern __shared__ __attribute__((aligned(16))) unsigned char shm[];
    LAS unsigned char* lds = (LAS unsigned char*)shm;
    const int tid = threadIdx.x, wid = __builtin_amdgcn_readfirstlane(tid >> 6), lane = tid & 63;
    const int b = blockIdx.x;
    char* myws = p.ws[p.rank];
    STAMP(0);
    const int wr = wid >> 2, wc = wid & 3;
    const int g = b & 7, idx = b >> 3;
    const int src = (p.rank + g) & (NRANK - 1);
    const int pm = 2 * src + (idx & 1), pn = idx >> 1;
    const int mem = ((g + pn) & 7) * 2 + (idx & 1);
    const char* wsrc = (const char*)(p.w + (size_t)(mem * 64 + (wid & 1) * 32) * N_ + pn * 256 + (wid >> 1) * 64);
    const unsigned wvoff = (unsigned)((lane >> 4) * 8 * N_ + (lane & 15) * 4) * 4u;
    char* bdst = myws + OFF_B16 + (size_t)((pn * 64 + mem) * 2) * HTB + ((wid >> 1) * 4 + (wid & 1)) * 1024;
    const unsigned lane16 = lane * 16;
    constexpr size_t WROUND = (size_t)1024 * N_ * 4, BROUND = (size_t)16 * 2 * HTB;
    unsigned* const tmo = (unsigned*)(myws + CB_TMO);
    unsigned long long bctr = (unsigned long long)myws + CB_BC + (pn * 32 + (mem >> 1)) * 128;
    asm volatile("" : "+s"(bctr));
#define WREGS "v224","v225","v226","v227","v228","v229","v230","v231","v232","v233","v234","v235","v236","v237","v238","v239","v240","v241","v242","v243","v244","v245","v246","v247","v248","v249","v250","v251","v252","v253","v254","v255"
#define LW1(R, r, j) asm volatile("global_load_dwordx4 " R ", %0, %1" WNT :: "v"(wvoff), "s"(wsrc + (size_t)(r) * WROUND + (size_t)(j) * N_ * 4) : "memory")
#define LOADW01(r) do { LW1("v[224:227]", r, 0); LW1("v[228:231]", r, 1); } while (0)
#define LOADW23(r) do { LW1("v[232:235]", r, 2); LW1("v[236:239]", r, 3); } while (0)
#define LOADW45(r) do { LW1("v[240:243]", r, 4); LW1("v[244:247]", r, 5); } while (0)
#define LOADW67(r) do { LW1("v[248:251]", r, 6); LW1("v[252:255]", r, 7); } while (0)
#define CV1(h, a0, a1, a2, a3, a4, a5, a6, a7) asm volatile("v_cvt_pk_f16_f32 %0, v" #a0 ", v" #a1 "\n\tv_cvt_pk_f16_f32 %1, v" #a2 ", v" #a3 "\n\tv_cvt_pk_f16_f32 %2, v" #a4 ", v" #a5 "\n\tv_cvt_pk_f16_f32 %3, v" #a6 ", v" #a7 \
        : "=v"(h[0]), "=v"(h[1]), "=v"(h[2]), "=v"(h[3]))
#define SCHED __builtin_amdgcn_sched_barrier(0)
#define STOREW01(r) do { const __amdgpu_buffer_rsrc_t _rs = __builtin_amdgcn_make_buffer_rsrc(bdst + (size_t)(r) * BROUND, 0, 4096, 0x00020000); u32x4 h0; u32x4 h1; \
        CV1(h0, 224, 228, 232, 236, 240, 244, 248, 252); CV1(h1, 225, 229, 233, 237, 241, 245, 249, 253); SCHED; \
        __builtin_amdgcn_raw_buffer_store_b128(h0, _rs, lane16, 0, 16); __builtin_amdgcn_raw_buffer_store_b128(h1, _rs, lane16 + 2048, 0, 16); SCHED; asm volatile("s_nop 1"); } while (0)
#define STOREW23(r) do { const __amdgpu_buffer_rsrc_t _rs = __builtin_amdgcn_make_buffer_rsrc(bdst + (size_t)(r) * BROUND + HTB, 0, 4096, 0x00020000); u32x4 h2; u32x4 h3; \
        CV1(h2, 226, 230, 234, 238, 242, 246, 250, 254); CV1(h3, 227, 231, 235, 239, 243, 247, 251, 255); SCHED; \
        __builtin_amdgcn_raw_buffer_store_b128(h2, _rs, lane16, 0, 16); __builtin_amdgcn_raw_buffer_store_b128(h3, _rs, lane16 + 2048, 0, 16); SCHED; asm volatile("s_nop 1"); } while (0)
#define SIGNALB(r) do { if (lane == 0) asm volatile("global_atomic_add %0, %1, %2 sc1" :: "v"(0u), "v"(1u), "s"(bctr + (r) * 1024) : "memory"); } while (0)
#define WAIT_V(n) asm volatile("s_waitcnt vmcnt(" #n ")" ::: "memory")
#define WAIT_L(n) asm volatile("s_waitcnt lgkmcnt(" #n ")" ::: "memory")
#define BAR __builtin_amdgcn_s_barrier()
    asm volatile("" ::: "v220", "v221", "v222", "v223", WREGS);

    const int jp = mem >> 1;
    const bool clsA = jp < 2, clsC = jp >= 5;
    const int xb = (7 - jp) * 32 + idx;
    const unsigned ldsw = __builtin_amdgcn_readfirstlane((unsigned)(size_t)lds + (unsigned)wid * 2048u);
    const unsigned pfoff = lane * 64;
    LAS unsigned* ctlw = (LAS unsigned*)(lds + 8 * HTB + 256);
    const char* fB = myws + CB_BC + pn * 32 * 128;
    const unsigned aflag4 = CB_AC + (src * 64 + (lane & 1) * 32 + ((lane >> 1) & 3)) * 128;
    const unsigned aflag64 = CB_AC + (src * 64 + lane) * 128;
    asm volatile("global_load_dword v220, %0, %1 sc1" :: "v"(0u), "s"(myws + CB_TK + b * 128) : "memory");
    unsigned call;
    const char* cA0;
#define CALL_FROM_V220() do { asm volatile("v_readfirstlane_b32 %0, v220" : "=s"(call)); call += 1u; cA0 = SRCWS + OFF_A16 + (size_t)(call & 1) * A16_BYTES; } while (0)
#define SRCWS p.ws[src]
#define SIGNALA(n) do { if (lane < NRANK) asm volatile("global_atomic_add v[222:223], %0, off sc1" :: "v"(n) : "memory"); } while (0)
#define LOADW0() do { LOADW01(0); LOADW23(0); LOADW45(0); LOADW67(0); } while (0)
#define PFO(off) asm volatile("s_mov_b32 m0, %2\n\ts_nop 0\n\tglobal_load_lds_dword %0, %1" :: "v"(pfoff), "s"(cA0 + (off)), "s"(ldsw - wid * 2048 + 8 * HTB) : "memory")
#define PF(q) PFO(idx * 4096 + (size_t)(q) * 131072)
#define DLY 32
    if (wid < 4) {
        f32x4 xv[8];
        asm volatile("global_load_dwordx2 v[222:223], %0, %1" :: "v"((lane & 7) * 8), "s"((const char*)__builtin_amdgcn_kernarg_segment_ptr() + __builtin_offsetof(Params, ws)) : "memory");
        if (tid == 0) { ctlw[0] = 0u; ctlw[1] = 0u; ctlw[2] = 0u; WAIT_L(0); }
        if (!clsC) { if (!clsA) __builtin_amdgcn_s_sleep(DLY); LOADW0(); }
        { const char* s = (const char*)(p.x + (size_t)((xb & 3) * 128 + wid * 32) * K_ + (xb >> 2) * 64);
          const unsigned xo = (unsigned)((lane & 15) * K_ + (lane >> 4) * 8) * 4u;
          asm volatile("global_load_dwordx4 %0, %4, %5\n\tglobal_load_dwordx4 %1, %4, %5 offset:16\n\tglobal_load_dwordx4 %2, %4, %5 offset:128\n\tglobal_load_dwordx4 %3, %4, %5 offset:144"
                       : "=&v"(xv[0]), "=&v"(xv[1]), "=&v"(xv[2]), "=&v"(xv[3]) : "v"(xo), "s"(s) : "memory");
          asm volatile("global_load_dwordx4 %0, %4, %5\n\tglobal_load_dwordx4 %1, %4, %5 offset:16\n\tglobal_load_dwordx4 %2, %4, %5 offset:128\n\tglobal_load_dwordx4 %3, %4, %5 offset:144"
                       : "=&v"(xv[4]), "=&v"(xv[5]), "=&v"(xv[6]), "=&v"(xv[7]) : "v"(xo), "s"(s + (size_t)16 * K_ * 4) : "memory"); }
        BAR;
        if (!clsC) { WAIT_V(8); STOREW01(0); STOREW23(0); WAIT_V(4); } else WAIT_V(0);
        asm volatile("" : "+v"(xv[0]), "+v"(xv[1]), "+v"(xv[2]), "+v"(xv[3]), "+v"(xv[4]), "+v"(xv[5]), "+v"(xv[6]), "+v"(xv[7]) :: "memory");
        CALL_FROM_V220();
        asm volatile("v_lshl_add_u64 v[222:223], v[222:223], 0, %0" :: "s"((unsigned long long)(CB_AC + ((p.rank * 2 + ((xb >> 1) & 1)) * 32 + (xb >> 3)) * 128)));
        { char* dst = myws + OFF_A16 + (size_t)(call & 1) * A16_BYTES + (size_t)xb * HTB + wid * 4096;
          const __amdgpu_buffer_rsrc_t rs = __builtin_amdgcn_make_buffer_rsrc(dst, 0, 4096, 0x00020000);
#pragma unroll
          for (int i = 0; i < 4; ++i) __builtin_amdgcn_raw_buffer_store_b128(__builtin_bit_cast(u32x4, cvt8(xv[2 * i], xv[2 * i + 1])), rs, lane16 + i * 1024, 0, 17); }
        SCHED;
        if (jp <= 2) { WAIT_V(4); SIGNALB(0); }
        if (clsC) {
            WAIT_V(0);
            unsigned arr = 0;
            if (lane == 0) arr = __hip_atomic_fetch_add(ctlw + 2, 1u, __ATOMIC_RELAXED, __HIP_MEMORY_SCOPE_WORKGROUP);
            if (__builtin_amdgcn_readfirstlane(arr) == 3u) SIGNALA(4u);
        }
        STAMP(7);
        if (wid == 0) {
            const bool to = wait_words((const GAS unsigned*)(fB + (lane & 1) * 128), 16u * call, tmo, call);
            if (to && lane == 0) ctlw[0] = 1u;
            WAIT_L(0);
        }
    } else {
        if (wid == 4) asm volatile("buffer_inv sc0 sc1" ::: "memory");
        if (!clsC) { if (!clsA) __builtin_amdgcn_s_sleep(DLY); LOADW0(); BAR; WAIT_V(0); STOREW01(0); STOREW23(0); if (jp <= 2) { WAIT_V(0); SIGNALB(0); } }
        else { WAIT_V(0); BAR; }
        CALL_FROM_V220();
        if (wid == 4 || jp == 5) {
            const bool to = wait_words((const GAS unsigned*)(myws + aflag4), 16u * call, tmo, call);
            if (to && lane == 0) ctlw[1] = 1u;
            WAIT_L(0);
            STAMPW(8, 4);
            if (jp == 5) {
                const unsigned u = (((pn >> 3) * 2 + (idx & 1)) * 4 + (wid - 4)) * 4096;
                PFO(u); WAIT_V(0); PFO(u + 65536); WAIT_V(0); PFO(u + 131072);
            }
        }
    }
    STAMP(1);
    BAR;
    asm volatile("" ::: "memory");
    if (tid == 0) __hip_atomic_store((unsigned*)(myws + CB_TK + b * 128), call, __ATOMIC_RELAXED, __HIP_MEMORY_SCOPE_AGENT);
    unsigned bad = 0;
    STAMP(6);
    const char* cA = cA0 + (pm & 1) * 2 * HTB;
    const char* cB = myws + OFF_B16 + (size_t)pn * 64 * 2 * HTB;
    const unsigned voff = wid * 2048 + lane * 16;
    const int aoff = wr * 8192 + lane * 16, boff = wc * 4096 + lane * 16;
    const unsigned fneed = 16u * call;
#define SA(b, h) (((b) * 2 + (h)) * HTB)
#define SB(b, h) ((4 + (b) * 2 + (h)) * HTB)
#define STAGE(bufoff, gbase, AUX) asm volatile("s_add_u32 m0, %1, %3\n\ts_nop 0\n\tglobal_load_lds_dwordx4 %0, %2" AUX "\n\tglobal_load_lds_dwordx4 %0, %2 offset:1024" AUX \
        :: "v"(voff), "s"(ldsw), "s"((const char*)(gbase)), "i"(bufoff) : "memory", "scc")
#define STA(b, h, kt) STAGE(SA(b, h), cA + (size_t)(kt) * 65536 + (h) * HTB, "")
#define STB(b, h, kt) STAGE(SB(b, h), cB + (size_t)(kt) * 32768 + (h) * HTB, " sc1")
#define LDA(dst, b, h) do { _Pragma("unroll") for (int m = 0; m < 4; ++m) _Pragma("unroll") for (int k = 0; k < 2; ++k) dst[m][k] = *(const LAS half8*)(lds + SA(b, h) + aoff + m * 2048 + k * 1024); } while (0)
#define LDB(dst, b, h) do { _Pragma("unroll") for (int n = 0; n < 2; ++n) _Pragma("unroll") for (int k = 0; k < 2; ++k) dst[n][k] = *(const LAS half8*)(lds + SB(b, h) + boff + n * 2048 + k * 1024); } while (0)
#define MMA(ai, bj, At, Bt) do { __builtin_amdgcn_s_setprio(1); _Pragma("unroll") for (int k = 0; k < 2; ++k) _Pragma("unroll") for (int mm = 0; mm < 4; ++mm) _Pragma("unroll") for (int nn = 0; nn < 2; ++nn) { \
        const int m = k ? 3 - mm : mm, n = (mm & 1) ? 1 - nn : nn; \
        asm volatile("v_mfma_f32_16x16x32_f16 %0, %1, %2, %0" : "+v"(acc[ai][bj][m][n]) : "v"(At[m][k]), "v"(Bt[n][k])); } __builtin_amdgcn_s_setprio(0); } while (0)
#define IST(t) do {} while (0)
#define FLAGLD(q) asm volatile("global_load_dword v220, %0, %1 sc0 sc1" :: "v"(0u), "s"(fB + (q) * 128) : "memory")
#define FLAGCK(q) do { unsigned fv; asm volatile("v_readfirstlane_b32 %0, v220" : "=s"(fv)); if ((int)(fv - fneed) < 0) { if (wait_words((const GAS unsigned*)(fB + (q) * 128), fneed, tmo, call)) bad = 1u; \
        asm volatile("s_waitcnt vmcnt(0)" ::: "memory"); } } while (0)
    f32x4 acc[2][2][4][2];
#pragma unroll
    for (int a = 0; a < 2; ++a)
#pragma unroll
        for (int bb = 0; bb < 2; ++bb)
#pragma unroll
            for (int m = 0; m < 4; ++m)
#pragma unroll
                for (int n = 0; n < 2; ++n) { acc[a][bb][m][n] = (f32x4){0.f, 0.f, 0.f, 0.f}; asm volatile("" : "+v"(acc[a][bb][m][n])); }
    half8 At[4][2], B0[2][2], B1[2][2];
    unsigned its = 0;
    STB(0, 0, 0); STA(0, 0, 0); STB(0, 1, 0); STA(0, 1, 0);
    if (wr == 1) BAR;
    WAIT_V(4); BAR;
    STB(1, 0, 1); STA(1, 0, 1); STB(1, 1, 1);
    WAIT_V(6); BAR;
    const int pstart = jp + (wid & 1) + SHIFT;
    unsigned mLd = 0x010101u << pstart;
    if (clsC) mLd |= 1u << (jp == 5 ? 0 : (wid & 1));
    unsigned mSt = mLd << 1, mSg = mLd << 2;
    if (jp == 3 || jp == 4) mSg |= 1u;
    unsigned mFl = wr == 0 ? 0x11111111u << wc : 0u;
    unsigned mPf = wr == 1 ? 0x11111111u << wc : 0u;
    unsigned mEv = wid == 4 ? 0x202u : (wid < 4 && !clsC) ? 1u : 0u;
    unsigned mAs = wid == 4 ? 0x101u : 0u;
    unsigned mLS = mLd | mSt;
#define RFL(x) x = __builtin_amdgcn_readfirstlane(x)
    RFL(mLd); RFL(mSt); RFL(mSg); RFL(mFl); RFL(mPf); RFL(mEv); RFL(mAs); RFL(mLS);
    asm volatile("" : "+s"(mLd), "+s"(mSg), "+s"(mFl), "+s"(mPf), "+s"(mEv), "+s"(mAs), "+s"(mLS));
#define BIT(m) UNL(((m) >> i) & 1u)
#define RND(off) (i - (off) < pstart ? 0 : ((i - (off) - pstart) >> 3) + 1)
#define Q2 (i + 2 < 32 ? i + 2 : 31)
    const char* pa = cA + 131072;
    const char* pb = cB + 65536;
    for (int i = 0; i < NT / 2; ++i) {
        IST(2 * i); SCHED;
        LDB(B0, 0, 0); SCHED; LDA(At, 0, 0); STAGE(SA(1, 1), pa - 65536 + HTB, "");
        WAIT_L(8); BAR; WAIT_L(0); MMA(0, 0, At, B0); BAR; SCHED;
        LDB(B1, 0, 1); STAGE(SB(0, 0), pb, " sc1"); if (BIT(mLd)) LOADW01(RND(0));
        BAR; WAIT_L(0); MMA(0, 1, At, B1); BAR; SCHED;
        LDA(At, 0, 1); STAGE(SA(0, 0), pa, ""); if (BIT(mLd)) LOADW23(RND(0));
        BAR; WAIT_L(0); MMA(1, 0, At, B0); BAR; SCHED;
        STAGE(SB(0, 1), pb + HTB, " sc1");
        if (BIT(mFl)) { FLAGLD(Q2); if (BIT(mLd)) WAIT_V(11); else WAIT_V(7); }
        else if (BIT(mLd)) WAIT_V(10); else WAIT_V(6);
        if (BIT(mSg)) SIGNALB(RND(2));
        if (BIT(mEv)) {
            if (wid < 4) SIGNALA(1u);
            else {
                const unsigned need = (i == 1 && (lane & 31) >= 12) ? 0u : fneed;
                unsigned av; asm volatile("v_mov_b32 %0, v221" : "=v"(av));
                if (!__all((int)(av - need) >= 0)) { if (wait_words((const GAS unsigned*)(myws + aflag64), need, tmo, call)) { if (lane == 0) ctlw[1] = 1u; } WAIT_V(0); }
            }
        }
        BAR; MMA(1, 1, At, B1); BAR; SCHED;
        LDB(B0, 1, 0); SCHED; LDA(At, 1, 0); STAGE(SA(0, 1), pa + HTB, "");
        WAIT_L(8); BAR; WAIT_L(0); MMA(0, 0, At, B0); BAR; SCHED;
        LDB(B1, 1, 1); STAGE(SB(1, 0), pb + 32768, " sc1");
        if (BIT(mPf)) PF(Q2);
        if (BIT(mLS)) { if (BIT(mLd)) LOADW45(RND(0)); else STOREW01(RND(1)); }
        BAR; WAIT_L(0); MMA(0, 1, At, B1); BAR; SCHED;
        LDA(At, 1, 1); STAGE(SA(1, 0), pa + 65536, "");
        if (BIT(mLS)) { if (BIT(mLd)) LOADW67(RND(0)); else STOREW23(RND(1)); }
        BAR; WAIT_L(0); MMA(1, 0, At, B0); BAR; SCHED;
        STAGE(SB(1, 1), pb + 32768 + HTB, " sc1");
        pa += 131072; pb += 65536;
        if (BIT(mPf)) { if (BIT(mLS)) WAIT_V(11); else WAIT_V(7); }
        else { if (BIT(mLS)) WAIT_V(10); else WAIT_V(6); if (BIT(mFl)) FLAGCK(Q2); }
        if (BIT(mAs)) asm volatile("global_load_dword v221, %0, %1 sc0 sc1" :: "v"(aflag64), "s"(myws) : "memory");
        BAR; MMA(1, 1, At, B1); BAR; SCHED;
    }
    if (wr == 0) { if (bad && lane == 0) ctlw[0] = 1u; WAIT_L(0); BAR; }
    BAR;
    asm volatile("s_nop 15\n\ts_nop 15" ::: "memory");
    bad |= __builtin_amdgcn_readfirstlane(*(volatile LAS unsigned*)ctlw | *(volatile LAS unsigned*)(ctlw + 1));
    SCHED;
    {
        int tz = threadIdx.x; asm volatile("" : "+v"(tz));
        const int fr = tz & 15, fq = (tz >> 4) & 3;
        float* o = p.out + (size_t)(pm * 256 + wr * 64 + fq * 4) * N_ + pn * 256 + wc * 64 + fr * 4;
        if (bad) {
            const float q = __builtin_nanf("");
#pragma unroll 1
            for (int i = 0; i < 32; ++i) *(f32x4*)(o + (size_t)((i >> 4) * 128 + ((i >> 2) & 3) * 16 + (i & 3)) * N_) = (f32x4){q, q, q, q};
        } else {
            const float k1 = -2.0f * 0.7978845608028654f * 1.4426950408889634f, k2 = k1 * 0.044715f;
            const __amdgpu_buffer_rsrc_t ors = __builtin_amdgcn_make_buffer_rsrc(p.out + (size_t)(pm * 256 + wr * 64) * N_ + pn * 256 + wc * 64, 0, 192 * N_ * 4, 0x00020000);
            const unsigned ovoff = (unsigned)(fq * 4 * N_ + fr * 4) * 4u;
#pragma unroll
            for (int ai = 0; ai < 2; ++ai)
#pragma unroll
                for (int m = 0; m < 4; ++m) {
                    f32x4 v[4];
#pragma unroll
                    for (int t = 0; t < 4; ++t)
#pragma unroll
                        for (int h = 0; h < 2; ++h) {
                            const f32x2 y = {acc[ai][t >> 1][m][t & 1][2 * h], acc[ai][t >> 1][m][t & 1][2 * h + 1]};
                            const f32x2 z = y * __builtin_elementwise_fma(y * y, (f32x2){k2, k2}, (f32x2){k1, k1});
                            f32x2 e = {__builtin_amdgcn_exp2f(z[0]), __builtin_amdgcn_exp2f(z[1])};
                            e += 1.0f;
                            float o0 = y[0] * __builtin_amdgcn_rcpf(e[0]), o1 = y[1] * __builtin_amdgcn_rcpf(e[1]);
                            asm("" : "+v"(o0)); asm("" : "+v"(o1));
                            v[2 * h][t] = o0;
                            v[2 * h + 1][t] = o1;
                        }
#pragma unroll
                    for (int j = 0; j < 4; ++j) __builtin_amdgcn_raw_buffer_store_b128(__builtin_bit_cast(u32x4, v[j]), ors, ovoff, (ai * 128 + m * 16 + j) * N_ * 4, OAUX);
                }
        }
    }
    STAMP(4);
    WAIT_V(0);
    STAMP(5);
}

extern "C" void kernel_launch(int rank, int nranks, ncclComm_t comm, hipStream_t stream,
                              void* const* d_in, const int* in_sizes, int n_in, void* d_out, int out_size,
                              const int* in_dtypes, int out_dtype, void* const* d_out_all, void* const* d_ws_all, size_t ws_bytes) {
    if (nranks != NRANK || ws_bytes < WS_NEED) { fprintf(stderr, "unsupported config\n"); return; }
    static bool init = false;
    if (!init) { (void)hipFuncSetAttribute((const void*)ag_gemm, hipFuncAttributeMaxDynamicSharedMemorySize, LDS_BYTES); init = true; }
    Params p; p.x = (const float*)d_in[0]; p.w = (const float*)d_in[1]; p.out = (float*)d_out; p.rank = rank; p.phase = 0;
    for (int q = 0; q < NRANK; ++q) p.ws[q] = (char*)d_ws_all[q];
    ag_gemm<<<256, 512, LDS_BYTES, stream>>>(p);
}
```

```cpp
#include <hip/hip_runtime.h>
#include <rccl/rccl.h>
#include <cstdio>

#define XNT " nt"
#define WNT " nt"
#define UNL(x) __builtin_expect(!!(x), 0)
#define LAS __attribute__((address_space(3)))
#define GAS __attribute__((address_space(1)))
typedef _Float16 half8 __attribute__((ext_vector_type(8)));
typedef float f32x4 __attribute__((ext_vector_type(4)));
typedef unsigned u32x4 __attribute__((ext_vector_type(4)));
typedef float f32x2 __attribute__((ext_vector_type(2)));
#define NO_EARLY
#define PF_NONE
#define SHIFT 0

constexpr int NRANK = 8;
constexpr int M_ = 4096, K_ = 4096, N_ = 4096, MS = M_ / NRANK;
constexpr int NT = K_ / 64;
constexpr int HTB = 16384;
constexpr int LDS_BYTES = 8 * HTB + 512;
constexpr size_t OFF_A16 = 1u << 20;
constexpr size_t A16_BYTES = (size_t)MS * K_ * 2;
constexpr size_t OFF_B16 = 16u << 20;
constexpr size_t WS_NEED = OFF_B16 + (size_t)K_ * N_ * 2 + (1u << 20);
constexpr unsigned long long DEADLINE = 50000000ull;
constexpr unsigned CB_TMO = 0;
constexpr unsigned CB_TK = 4096;
constexpr unsigned CB_BC = CB_TK + 256 * 128;
constexpr unsigned CB_AC = CB_BC + 512 * 128;

#define STAMPW(i, w) do {} while (0)
#define STAMP(i) STAMPW(i, 0)
#define OAUX 2
#define OSTORE(v, ptr) __builtin_nontemporal_store(v, ptr)
struct Params { const float* x; const float* w; float* out; char* ws[NRANK]; int rank; int phase; };

__device__ __forceinline__ half8 cvt8(f32x4 a, f32x4 b) {
    half8 h; h[0] = (_Float16)a[0]; h[1] = (_Float16)a[1]; h[2] = (_Float16)a[2]; h[3] = (_Float16)a[3];
    h[4] = (_Float16)b[0]; h[5] = (_Float16)b[1]; h[6] = (_Float16)b[2]; h[7] = (_Float16)b[3]; return h;
}
__device__ __forceinline__ float gelu1(float y) {
    const float k1 = -2.0f * 0.7978845608028654f * 1.4426950408889634f, k2 = k1 * 0.044715f;
    float z = y * __builtin_fmaf(y * y, k2, k1);
    float e = __builtin_amdgcn_exp2f(z);
    return y * __builtin_amdgcn_rcpf(1.0f + e);
}
__device__ __forceinline__ bool wait_words(const GAS unsigned* word, unsigned need, unsigned* tmo, unsigned call) {
    const unsigned long long t0 = __builtin_amdgcn_s_memrealtime();
    for (unsigned n = 1;; ++n) {
        const unsigned v = __hip_atomic_load(word, __ATOMIC_RELAXED, __HIP_MEMORY_SCOPE_SYSTEM);
        if (__all((int)(v - need) >= 0)) return false;
        if ((n & 15u) == 0u) {
            if (__builtin_amdgcn_readfirstlane(__hip_atomic_load(tmo, __ATOMIC_RELAXED, __HIP_MEMORY_SCOPE_AGENT)) == call || __builtin_amdgcn_s_memrealtime() - t0 > DEADLINE) {
                __hip_atomic_store(tmo, call, __ATOMIC_RELAXED, __HIP_MEMORY_SCOPE_AGENT); return true; }
        }
        __builtin_amdgcn_s_sleep(1);
    }
}

__global__ __launch_bounds__(512, 2) __attribute__((amdgpu_num_vgpr(110))) void ag_gemm(Params p) {
    extern __shared__ __attribute__((aligned(16))) unsigned char shm[];
    LAS unsigned char* lds = (LAS unsigned char*)shm;
    const int tid = threadIdx.x, wid = __builtin_amdgcn_readfirstlane(tid >> 6), lane = tid & 63;
    const int b = blockIdx.x;
    char* myws = p.ws[p.rank];
    STAMP(0);
    const int wr = wid >> 2, wc = wid & 3;
    const int g = b & 7, idx = b >> 3;
    const int src = (p.rank + g) & (NRANK - 1);
    const int pm = 2 * src + (idx & 1), pn = idx >> 1;
    const int mem = ((g + pn) & 7) * 2 + (idx & 1);
    const char* wsrc = (const char*)(p.w + (size_t)(mem * 64 + (wid & 1) * 32) * N_ + pn * 256 + (wid >> 1) * 64);
    const unsigned wvoff = (unsigned)((lane >> 4) * 8 * N_ + (lane & 15) * 4) * 4u;
    char* bdst = myws + OFF_B16 + (size_t)((pn * 64 + mem) * 2) * HTB + ((wid >> 1) * 4 + (wid & 1)) * 1024;
    const unsigned lane16 = lane * 16;
    constexpr size_t WROUND = (size_t)1024 * N_ * 4, BROUND = (size_t)16 * 2 * HTB;
    unsigned* const tmo = (unsigned*)(myws + CB_TMO);
    unsigned long long bctr = (unsigned long long)myws + CB_BC + (pn * 32 + (mem >> 1)) * 128;
    asm volatile("" : "+s"(bctr));
#define WREGS "v224","v225","v226","v227","v228","v229","v230","v231","v232","v233","v234","v235","v236","v237","v238","v239","v240","v241","v242","v243","v244","v245","v246","v247","v248","v249","v250","v251","v252","v253","v254","v255"
#define LW1(R, r, j) asm volatile("global_load_dwordx4 " R ", %0, %1" WNT :: "v"(wvoff), "s"(wsrc + (size_t)(r) * WROUND + (size_t)(j) * N_ * 4) : "memory")
#define LOADW01(r) do { LW1("v[224:227]", r, 0); LW1("v[228:231]", r, 1); } while (0)
#define LOADW23(r) do { LW1("v[232:235]", r, 2); LW1("v[236:239]", r, 3); } while (0)
#define LOADW45(r) do { LW1("v[240:243]", r, 4); LW1("v[244:247]", r, 5); } while (0)
#define LOADW67(r) do { LW1("v[248:251]", r, 6); LW1("v[252:255]", r, 7); } while (0)
#define CV1(h, a0, a1, a2, a3, a4, a5, a6, a7) asm volatile("v_cvt_pk_f16_f32 %0, v" #a0 ", v" #a1 "\n\tv_cvt_pk_f16_f32 %1, v" #a2 ", v" #a3 "\n\tv_cvt_pk_f16_f32 %2, v" #a4 ", v" #a5 "\n\tv_cvt_pk_f16_f32 %3, v" #a6 ", v" #a7 \
        : "=v"(h[0]), "=v"(h[1]), "=v"(h[2]), "=v"(h[3]))
#define SCHED __builtin_amdgcn_sched_barrier(0)
#define STOREW01(r) do { const __amdgpu_buffer_rsrc_t _rs = __builtin_amdgcn_make_buffer_rsrc(bdst + (size_t)(r) * BROUND, 0, 4096, 0x00020000); u32x4 h0; u32x4 h1; \
        CV1(h0, 224, 228, 232, 236, 240, 244, 248, 252); CV1(h1, 225, 229, 233, 237, 241, 245, 249, 253); SCHED; \
        __builtin_amdgcn_raw_buffer_store_b128(h0, _rs, lane16, 0, 16); __builtin_amdgcn_raw_buffer_store_b128(h1, _rs, lane16 + 2048, 0, 16); SCHED; asm volatile("s_nop 1"); } while (0)
#define STOREW23(r) do { const __amdgpu_buffer_rsrc_t _rs = __builtin_amdgcn_make_buffer_rsrc(bdst + (size_t)(r) * BROUND + HTB, 0, 4096, 0x00020000); u32x4 h2; u32x4 h3; \
        CV1(h2, 226, 230, 234, 238, 242, 246, 250, 254); CV1(h3, 227, 231, 235, 239, 243, 247, 251, 255); SCHED; \
        __builtin_amdgcn_raw_buffer_store_b128(h2, _rs, lane16, 0, 16); __builtin_amdgcn_raw_buffer_store_b128(h3, _rs, lane16 + 2048, 0, 16); SCHED; asm volatile("s_nop 1"); } while (0)
#define SIGNALB(r) do { if (lane == 0) asm volatile("global_atomic_add %0, %1, %2 sc1" :: "v"(0u), "v"(1u), "s"(bctr + (r) * 1024) : "memory"); } while (0)
#define WAIT_V(n) asm volatile("s_waitcnt vmcnt(" #n ")" ::: "memory")
#define WAIT_L(n) asm volatile("s_waitcnt lgkmcnt(" #n ")" ::: "memory")
#define BAR __builtin_amdgcn_s_barrier()
    asm volatile("" ::: "v220", "v221", "v222", "v223", WREGS);

    const int jp = mem >> 1;
    const bool clsA = jp < 2, clsC = jp >= 5;
    const int xb = (7 - jp) * 32 + idx;
    const unsigned ldsw = __builtin_amdgcn_readfirstlane((unsigned)(size_t)lds + (unsigned)wid * 2048u);
    const unsigned pfoff = lane * 64;
    LAS unsigned* ctlw = (LAS unsigned*)(lds + 8 * HTB + 256);
    const char* fB = myws + CB_BC + pn * 32 * 128;
    const unsigned aflag4 = CB_AC + (src * 64 + (lane & 1) * 32 + ((lane >> 1) & 3)) * 128;
    const unsigned aflag64 = CB_AC + (src * 64 + lane) * 128;
    asm volatile("global_load_dword v220, %0, %1 sc1" :: "v"(0u), "s"(myws + CB_TK + b * 128) : "memory");
    unsigned call;
    const char* cA0;
#define CALL_FROM_V220() do { asm volatile("v_readfirstlane_b32 %0, v220" : "=s"(call)); call += 1u; cA0 = SRCWS + OFF_A16 + (size_t)(call & 1) * A16_BYTES; } while (0)
#define SRCWS p.ws[src]
#define SIGNALA(n) do { if (lane < NRANK) asm volatile("global_atomic_add v[222:223], %0, off sc1" :: "v"(n) : "memory"); } while (0)
#define LOADW0() do { LOADW01(0); LOADW23(0); LOADW45(0); LOADW67(0); } while (0)
#define PFO(off) asm volatile("s_mov_b32 m0, %2\n\ts_nop 0\n\tglobal_load_lds_dword %0, %1" :: "v"(pfoff), "s"(cA0 + (off)), "s"(ldsw - wid * 2048 + 8 * HTB) : "memory")
#define PF(q) PFO(idx * 4096 + (size_t)(q) * 131072)
#define DLY 32
    if (wid < 4) {
        f32x4 xv[8];
        asm volatile("global_load_dwordx2 v[222:223], %0, %1" :: "v"((lane & 7) * 8), "s"((const char*)__builtin_amdgcn_kernarg_segment_ptr() + __builtin_offsetof(Params, ws)) : "memory");
        if (tid == 0) { ctlw[0] = 0u; ctlw[1] = 0u; ctlw[2] = 0u; WAIT_L(0); }
        if (!clsC) { if (!clsA) __builtin_amdgcn_s_sleep(DLY); LOADW0(); }
        { const char* s = (const char*)(p.x + (size_t)((xb & 3) * 128 + wid * 32) * K_ + (xb >> 2) * 64);
          const unsigned xo = (unsigned)((lane & 15) * K_ + (lane >> 4) * 8) * 4u;
          asm volatile("global_load_dwordx4 %0, %4, %5" XNT "\n\tglobal_load_dwordx4 %1, %4, %5 offset:16" XNT "\n\tglobal_load_dwordx4 %2, %4, %5 offset:128" XNT "\n\tglobal_load_dwordx4 %3, %4, %5 offset:144" XNT
                       : "=&v"(xv[0]), "=&v"(xv[1]), "=&v"(xv[2]), "=&v"(xv[3]) : "v"(xo), "s"(s) : "memory");
          asm volatile("global_load_dwordx4 %0, %4, %5" XNT "\n\tglobal_load_dwordx4 %1, %4, %5 offset:16" XNT "\n\tglobal_load_dwordx4 %2, %4, %5 offset:128" XNT "\n\tglobal_load_dwordx4 %3, %4, %5 offset:144" XNT
                       : "=&v"(xv[4]), "=&v"(xv[5]), "=&v"(xv[6]), "=&v"(xv[7]) : "v"(xo), "s"(s + (size_t)16 * K_ * 4) : "memory"); }
        BAR;
        if (!clsC) { WAIT_V(8); STOREW01(0); STOREW23(0); WAIT_V(4); } else WAIT_V(0);
        asm volatile("" : "+v"(xv[0]), "+v"(xv[1]), "+v"(xv[2]), "+v"(xv[3]), "+v"(xv[4]), "+v"(xv[5]), "+v"(xv[6]), "+v"(xv[7]) :: "memory");
        CALL_FROM_V220();
        asm volatile("v_lshl_add_u64 v[222:223], v[222:223], 0, %0" :: "s"((unsigned long long)(CB_AC + ((p.rank * 2 + ((xb >> 1) & 1)) * 32 + (xb >> 3)) * 128)));
        { char* dst = myws + OFF_A16 + (size_t)(call & 1) * A16_BYTES + (size_t)xb * HTB + wid * 4096;
          const __amdgpu_buffer_rsrc_t rs = __builtin_amdgcn_make_buffer_rsrc(dst, 0, 4096, 0x00020000);
#pragma unroll
          for (int i = 0; i < 4; ++i) __builtin_amdgcn_raw_buffer_store_b128(__builtin_bit_cast(u32x4, cvt8(xv[2 * i], xv[2 * i + 1])), rs, lane16 + i * 1024, 0, 17); }
        SCHED;
        if (jp <= 2) { WAIT_V(4); SIGNALB(0); }
        if (clsC) {
            WAIT_V(0);
            unsigned arr = 0;
            if (lane == 0) arr = __hip_atomic_fetch_add(ctlw + 2, 1u, __ATOMIC_RELAXED, __HIP_MEMORY_SCOPE_WORKGROUP);
            if (__builtin_amdgcn_readfirstlane(arr) == 3u) SIGNALA(4u);
        }
        STAMP(7);
        if (wid == 0) {
            const bool to = wait_words((const GAS unsigned*)(fB + (lane & 1) * 128), 16u * call, tmo, call);
            if (to && lane == 0) ctlw[0] = 1u;
            WAIT_L(0);
        }
    } else {
        if (wid == 4) asm volatile("buffer_inv sc0 sc1" ::: "memory");
        if (!clsC) { if (!clsA) __builtin_amdgcn_s_sleep(DLY); LOADW0(); BAR; WAIT_V(0); STOREW01(0); STOREW23(0); if (jp <= 2) { WAIT_V(0); SIGNALB(0); } }
        else { WAIT_V(0); BAR; }
        CALL_FROM_V220();
        if (wid == 4 || jp == 5) {
            const bool to = wait_words((const GAS unsigned*)(myws + aflag4), 16u * call, tmo, call);
            if (to && lane == 0) ctlw[1] = 1u;
            WAIT_L(0);
            STAMPW(8, 4);
            if (jp == 5) {
                const unsigned u = (((pn >> 3) * 2 + (idx & 1)) * 4 + (wid - 4)) * 4096;
                PFO(u); WAIT_V(0); PFO(u + 65536); WAIT_V(0); PFO(u + 131072);
            }
        }
    }
    STAMP(1);
    BAR;
    asm volatile("" ::: "memory");
    if (tid == 0) __hip_atomic_store((unsigned*)(myws + CB_TK + b * 128), call, __ATOMIC_RELAXED, __HIP_MEMORY_SCOPE_AGENT);
    unsigned bad = 0;
    STAMP(6);
    const char* cA = cA0 + (pm & 1) * 2 * HTB;
    const char* cB = myws + OFF_B16 + (size_t)pn * 64 * 2 * HTB;
    const unsigned voff = wid * 2048 + lane * 16;
    const int aoff = wr * 8192 + lane * 16, boff = wc * 4096 + lane * 16;
    const unsigned fneed = 16u * call;
#define SA(b, h) (((b) * 2 + (h)) * HTB)
#define SB(b, h) ((4 + (b) * 2 + (h)) * HTB)
#define STAGE(bufoff, gbase, AUX) asm volatile("s_add_u32 m0, %1, %3\n\ts_nop 0\n\tglobal_load_lds_dwordx4 %0, %2" AUX "\n\tglobal_load_lds_dwordx4 %0, %2 offset:1024" AUX \
        :: "v"(voff), "s"(ldsw), "s"((const char*)(gbase)), "i"(bufoff) : "memory", "scc")
#define STA(b, h, kt) STAGE(SA(b, h), cA + (size_t)(kt) * 65536 + (h) * HTB, "")
#define STB(b, h, kt) STAGE(SB(b, h), cB + (size_t)(kt) * 32768 + (h) * HTB, " sc1")
#define LDA(dst, b, h) do { _Pragma("unroll") for (int m = 0; m < 4; ++m) _Pragma("unroll") for (int k = 0; k < 2; ++k) dst[m][k] = *(const LAS half8*)(lds + SA(b, h) + aoff + m * 2048 + k * 1024); } while (0)
#define LDB(dst, b, h) do { _Pragma("unroll") for (int n = 0; n < 2; ++n) _Pragma("unroll") for (int k = 0; k < 2; ++k) dst[n][k] = *(const LAS half8*)(lds + SB(b, h) + boff + n * 2048 + k * 1024); } while (0)
#define MMA(ai, bj, At, Bt) do { __builtin_amdgcn_s_setprio(1); _Pragma("unroll") for (int k = 0; k < 2; ++k) _Pragma("unroll") for (int mm = 0; mm < 4; ++mm) _Pragma("unroll") for (int nn = 0; nn < 2; ++nn) { \
        const int m = k ? 3 - mm : mm, n = (mm & 1) ? 1 - nn : nn; \
        asm volatile("v_mfma_f32_16x16x32_f16 %0, %1, %2, %0" : "+v"(acc[ai][bj][m][n]) : "v"(At[m][k]), "v"(Bt[n][k])); } __builtin_amdgcn_s_setprio(0); } while (0)
#define IST(t) do {} while (0)
#define FLAGLD(q) asm volatile("global_load_dword v220, %0, %1 sc0 sc1" :: "v"(0u), "s"(fB + (q) * 128) : "memory")
#define FLAGCK(q) do { unsigned fv; asm volatile("v_readfirstlane_b32 %0, v220" : "=s"(fv)); if ((int)(fv - fneed) < 0) { if (wait_words((const GAS unsigned*)(fB + (q) * 128), fneed, tmo, call)) bad = 1u; \
        asm volatile("s_waitcnt vmcnt(0)" ::: "memory"); } } while (0)
    f32x4 acc[2][2][4][2];
#pragma unroll
    for (int a = 0; a < 2; ++a)
#pragma unroll
        for (int bb = 0; bb < 2; ++bb)
#pragma unroll
            for (int m = 0; m < 4; ++m)
#pragma unroll
                for (int n = 0; n < 2; ++n) { acc[a][bb][m][n] = (f32x4){0.f, 0.f, 0.f, 0.f}; asm volatile("" : "+v"(acc[a][bb][m][n])); }
    half8 At[4][2], B0[2][2], B1[2][2];
    unsigned its = 0, itr = 0;
    STB(0, 0, 0); STA(0, 0, 0); STB(0, 1, 0); STA(0, 1, 0);
    if (wr == 1) BAR;
    WAIT_V(4); BAR;
    STB(1, 0, 1); STA(1, 0, 1); STB(1, 1, 1);
    WAIT_V(6); BAR;
    const int pstart = jp + (wid & 1) + SHIFT;
    unsigned mLd = 0x010101u << pstart;
    if (clsC) mLd |= 1u << (jp == 5 ? 0 : (wid & 1));
    unsigned mSt = mLd << 1, mSg = mLd << 2;
    if (jp == 3 || jp == 4) mSg |= 1u;
    unsigned mFl = wr == 0 ? 0x11111111u << wc : 0u;
    unsigned mPf = wr == 1 ? 0x11111111u << wc : 0u;
    unsigned mEv = wid == 4 ? 0x202u : (wid < 4 && !clsC) ? 1u : 0u;
    unsigned mAs = wid == 4 ? 0x101u : 0u;
    unsigned mLS = mLd | mSt;
#define RFL(x) x = __builtin_amdgcn_readfirstlane(x)
    RFL(mLd); RFL(mSt); RFL(mSg); RFL(mFl); RFL(mPf); RFL(mEv); RFL(mAs); RFL(mLS);
    asm volatile("" : "+s"(mLd), "+s"(mSg), "+s"(mFl), "+s"(mPf), "+s"(mEv), "+s"(mAs), "+s"(mLS));
#define BIT(m) UNL(((m) >> i) & 1u)
#define RND(off) (i - (off) < pstart ? 0 : ((i - (off) - pstart) >> 3) + 1)
#define Q2 (i + 2 < 32 ? i + 2 : 31)
    const char* pa = cA + 131072;
    const char* pb = cB + 65536;
    constexpr int NIT = NT / 2 - 1;
    for (int i = 0; i < NIT; ++i) {
        IST(2 * i); SCHED;
        LDB(B0, 0, 0); SCHED; LDA(At, 0, 0); STAGE(SA(1, 1), pa - 65536 + HTB, "");
        WAIT_L(8); BAR; WAIT_L(0); MMA(0, 0, At, B0); BAR; SCHED;
        LDB(B1, 0, 1); STAGE(SB(0, 0), pb, " sc1"); if (BIT(mLd)) LOADW01(RND(0));
        BAR; WAIT_L(0); MMA(0, 1, At, B1); BAR; SCHED;
        LDA(At, 0, 1); STAGE(SA(0, 0), pa, ""); if (BIT(mLd)) LOADW23(RND(0));
        BAR; WAIT_L(0); MMA(1, 0, At, B0); BAR; SCHED;
        STAGE(SB(0, 1), pb + HTB, " sc1");
        if (BIT(mFl)) { FLAGLD(Q2); if (BIT(mLd)) WAIT_V(11); else WAIT_V(7); }
        else if (BIT(mLd)) WAIT_V(10); else WAIT_V(6);
        if (BIT(mSg)) SIGNALB(RND(2));
        if (BIT(mEv)) {
            if (wid < 4) SIGNALA(1u);
            else {
                const unsigned need = (i == 1 && (lane & 31) >= 12) ? 0u : fneed;
                unsigned av; asm volatile("v_mov_b32 %0, v221" : "=v"(av));
                if (!__all((int)(av - need) >= 0)) { if (wait_words((const GAS unsigned*)(myws + aflag64), need, tmo, call)) { if (lane == 0) ctlw[1] = 1u; } WAIT_V(0); }
            }
        }
        BAR; MMA(1, 1, At, B1); BAR; SCHED;
        LDB(B0, 1, 0); SCHED; LDA(At, 1, 0); STAGE(SA(0, 1), pa + HTB, "");
        WAIT_L(8); BAR; WAIT_L(0); MMA(0, 0, At, B0); BAR; SCHED;
        LDB(B1, 1, 1); STAGE(SB(1, 0), pb + 32768, " sc1");
        if (BIT(mPf)) PF(Q2);
        if (BIT(mLS)) { if (BIT(mLd)) LOADW45(RND(0)); else STOREW01(RND(1)); }
        BAR; WAIT_L(0); MMA(0, 1, At, B1); BAR; SCHED;
        LDA(At, 1, 1); STAGE(SA(1, 0), pa + 65536, "");
        if (BIT(mLS)) { if (BIT(mLd)) LOADW67(RND(0)); else STOREW23(RND(1)); }
        BAR; WAIT_L(0); MMA(1, 0, At, B0); BAR; SCHED;
        STAGE(SB(1, 1), pb + 32768 + HTB, " sc1");
        pa += 131072; pb += 65536;
        if (BIT(mPf)) { if (BIT(mLS)) WAIT_V(11); else WAIT_V(7); }
        else { if (BIT(mLS)) WAIT_V(10); else WAIT_V(6); if (BIT(mFl)) FLAGCK(Q2); }
        if (BIT(mAs)) asm volatile("global_load_dword v221, %0, %1 sc0 sc1" :: "v"(aflag64), "s"(myws) : "memory");
        BAR; MMA(1, 1, At, B1); BAR; SCHED;
    }
    {
        IST(62); SCHED;
        half8 Aa[2][2], Ab[2][2];
#define LDA1(dst, kt, ai, m) do { _Pragma("unroll") for (int k = 0; k < 2; ++k) dst[kt][k] = *(const LAS half8*)(lds + SA(kt, ai) + aoff + (m) * 2048 + k * 1024); } while (0)
#define MF1(ai, bj, m, n, A, B) asm volatile("v_mfma_f32_16x16x32_f16 %0, %1, %2, %0" : "+v"(acc[ai][bj][m][n]) : "v"(A), "v"(B))
#define MMH(ai, m, A, Bx, By) do { _Pragma("unroll") for (int k = 0; k < 2; ++k) { MF1(ai, 0, m, 0, A[k], Bx[0][k]); MF1(ai, 0, m, 1, A[k], Bx[1][k]); MF1(ai, 1, m, 0, A[k], By[0][k]); MF1(ai, 1, m, 1, A[k], By[1][k]); } } while (0)
#define MF2(ai, bj, m, n, A, R) asm volatile("v_mfma_f32_16x16x32_f16 %0, %1, " R ", %0" : "+v"(acc[ai][bj][m][n]) : "v"(A))
#define MMH2(ai, m, A) do { MF2(ai, 0, m, 0, A[0], "v[224:227]"); MF2(ai, 0, m, 1, A[0], "v[232:235]"); MF2(ai, 1, m, 0, A[0], "v[240:243]"); MF2(ai, 1, m, 1, A[0], "v[248:251]"); \
                            MF2(ai, 0, m, 0, A[1], "v[228:231]"); MF2(ai, 0, m, 1, A[1], "v[236:239]"); MF2(ai, 1, m, 0, A[1], "v[244:247]"); MF2(ai, 1, m, 1, A[1], "v[252:255]"); } while (0)
#define MMB(ai, m, A) do { __builtin_amdgcn_s_setprio(1); MMH(ai, m, A[0], B0, B1); MMH2(ai, m, A[1]); __builtin_amdgcn_s_setprio(0); } while (0)
        LDA1(Aa, 0, 0, 0); LDB(B0, 0, 0); LDB(B1, 0, 1); STAGE(SA(1, 1), pa - 65536 + HTB, "");
        if (UNL(bad) && lane == 0) ctlw[0] = 1u;
        if (wr == 0) BAR;
        WAIT_L(0); MMH(0, 0, Aa[0], B0, B1); SCHED;
        WAIT_V(2); BAR; SCHED;
        asm volatile("ds_read_b128 v[224:227], %0\n\tds_read_b128 v[228:231], %0 offset:1024\n\tds_read_b128 v[232:235], %0 offset:2048\n\tds_read_b128 v[236:239], %0 offset:3072\n\t"
                     "ds_read_b128 v[240:243], %0 offset:16384\n\tds_read_b128 v[244:247], %0 offset:17408\n\tds_read_b128 v[248:251], %0 offset:18432\n\tds_read_b128 v[252:255], %0 offset:19456"
                     :: "v"((unsigned)(size_t)lds + SB(1, 0) + boff) : "memory");
        LDA1(Aa, 1, 0, 0);
        bad = *(volatile LAS unsigned*)ctlw | *(volatile LAS unsigned*)(ctlw + 1);
        WAIT_L(0); MMH2(0, 0, Aa[1]); SCHED;
        LDA1(Aa, 0, 0, 1); LDA1(Aa, 1, 0, 1);
        bad = __builtin_amdgcn_readfirstlane(bad);
        int tz = threadIdx.x; asm volatile("" : "+v"(tz));
        const unsigned ovoff = (unsigned)(((tz >> 4) & 3) * 4 * N_ + (tz & 15) * 4) * 4u;
        const __amdgpu_buffer_rsrc_t ors = __builtin_amdgcn_make_buffer_rsrc(p.out + (size_t)(pm * 256 + wr * 64) * N_ + pn * 256 + wc * 64, 0, 192 * N_ * 4, 0x00020000);
#define EPI(ai, m) do { const float k1 = -2.0f * 0.7978845608028654f * 1.4426950408889634f, k2 = k1 * 0.044715f; f32x4 v[4]; \
            _Pragma("unroll") for (int t = 0; t < 4; ++t) _Pragma("unroll") for (int h = 0; h < 2; ++h) { \
                const f32x2 y = {acc[ai][t >> 1][m][t & 1][2 * h], acc[ai][t >> 1][m][t & 1][2 * h + 1]}; \
                const f32x2 z = y * __builtin_elementwise_fma(y * y, (f32x2){k2, k2}, (f32x2){k1, k1}); \
                f32x2 e = {__builtin_amdgcn_exp2f(z[0]), __builtin_amdgcn_exp2f(z[1])}; e += 1.0f; \
                float o0 = y[0] * __builtin_amdgcn_rcpf(e[0]); float o1 = y[1] * __builtin_amdgcn_rcpf(e[1]); \
                asm("" : "+v"(o0)); asm("" : "+v"(o1)); v[2 * h][t] = o0; v[2 * h + 1][t] = o1; } \
            _Pragma("unroll") for (int j = 0; j < 4; ++j) __builtin_amdgcn_raw_buffer_store_b128(__builtin_bit_cast(u32x4, v[j]), ors, ovoff, ((ai) * 128 + (m) * 16 + j) * N_ * 4, OAUX); } while (0)
        WAIT_V(0); BAR; SCHED;
        if (UNL(bad)) {
            const unsigned q = 0x7fc00000u;
#pragma unroll 1
            for (int i = 0; i < 32; ++i) __builtin_amdgcn_raw_buffer_store_b128((u32x4){q, q, q, q}, ors, ovoff, ((i >> 4) * 128 + ((i >> 2) & 3) * 16 + (i & 3)) * N_ * 4, 0);
        } else {
            STAMP(3);
#define PADM asm volatile("s_nop 7" ::: "memory")
            WAIT_L(0); MMB(0, 1, Aa); SCHED;
            EPI(0, 0); SCHED; LDA1(Aa, 0, 0, 2); LDA1(Aa, 1, 0, 2); LDA1(Ab, 0, 0, 3); LDA1(Ab, 1, 0, 3); SCHED;
            WAIT_L(4); MMB(0, 2, Aa); SCHED;
            LDA1(Aa, 0, 1, 0); LDA1(Aa, 1, 1, 0); SCHED; EPI(0, 1); SCHED;
            MMB(0, 3, Ab); SCHED;
            LDA1(Ab, 0, 1, 1); LDA1(Ab, 1, 1, 1); SCHED; EPI(0, 2); SCHED;
            MMB(1, 0, Aa); SCHED;
            LDA1(Aa, 0, 1, 2); LDA1(Aa, 1, 1, 2); SCHED; EPI(0, 3); SCHED;
            MMB(1, 1, Ab); SCHED;
            LDA1(Ab, 0, 1, 3); LDA1(Ab, 1, 1, 3); SCHED; EPI(1, 0); SCHED;
            MMB(1, 2, Aa); SCHED;
            EPI(1, 1); SCHED;
            MMB(1, 3, Ab); SCHED;
            EPI(1, 2); SCHED;
            PADM; SCHED;
            EPI(1, 3);
        }
    }
    STAMP(4);
    WAIT_V(0);
    STAMP(5);
}

extern "C" void kernel_launch(int rank, int nranks, ncclComm_t comm, hipStream_t stream,
                              void* const* d_in, const int* in_sizes, int n_in, void* d_out, int out_size,
                              const int* in_dtypes, int out_dtype, void* const* d_out_all, void* const* d_ws_all, size_t ws_bytes) {
    if (nranks != NRANK || ws_bytes < WS_NEED) { fprintf(stderr, "unsupported config\n"); return; }
    static bool init = false;
    if (!init) { (void)hipFuncSetAttribute((const void*)ag_gemm, hipFuncAttributeMaxDynamicSharedMemorySize, LDS_BYTES); init = true; }
    Params p; p.x = (const float*)d_in[0]; p.w = (const float*)d_in[1]; p.out = (float*)d_out; p.rank = rank; static int ncall = 0; p.phase = ncall++ > 0;
    const size_t wso = 0;
    for (int q = 0; q < NRANK; ++q) p.ws[q] = (char*)d_ws_all[q] + wso;
    ag_gemm<<<256, 512, LDS_BYTES, stream>>>(p);
}
```

```cpp
#include <hip/hip_runtime.h>
#include <rccl/rccl.h>
#include <cstdio>

#define UNL(x) __builtin_expect(!!(x), 0)
#define LAS __attribute__((address_space(3)))
#define GAS __attribute__((address_space(1)))
typedef _Float16 half8 __attribute__((ext_vector_type(8)));
typedef float f32x4 __attribute__((ext_vector_type(4)));
typedef float f32x2 __attribute__((ext_vector_type(2)));
typedef unsigned u32x4 __attribute__((ext_vector_type(4)));

constexpr int NRANK = 8;
constexpr int M_ = 4096, K_ = 4096, N_ = 4096, MS = M_ / NRANK;
constexpr int NT = K_ / 64;
constexpr int HTB = 16384;
constexpr int LDS_BYTES = 8 * HTB + 512;
constexpr size_t OFF_A16 = 1u << 20;
constexpr size_t A16_BYTES = (size_t)MS * K_ * 2;
constexpr size_t OFF_B16 = 16u << 20;
constexpr size_t WS_NEED = OFF_B16 + (size_t)K_ * N_ * 2;
constexpr unsigned long long DEADLINE = 50000000ull;
constexpr unsigned CB_TMO = 0;
constexpr unsigned CB_TK = 4096;
constexpr unsigned CB_BC = CB_TK + 256 * 128;
constexpr unsigned CB_AC = CB_BC + 512 * 128;
static_assert(CB_AC + 512 * 128 <= OFF_A16 && OFF_A16 + 2 * A16_BYTES <= OFF_B16, "workspace layout");

#define STAMPW(i, w) do {} while (0)
#define STAMP(i) STAMPW(i, 0)

struct Params { const float* x; const float* w; float* out; char* ws[NRANK]; int rank; };

__device__ __forceinline__ u32x4 cvt8(f32x4 a, f32x4 b) {
    half8 h; h[0] = (_Float16)a[0]; h[1] = (_Float16)a[1]; h[2] = (_Float16)a[2]; h[3] = (_Float16)a[3];
    h[4] = (_Float16)b[0]; h[5] = (_Float16)b[1]; h[6] = (_Float16)b[2]; h[7] = (_Float16)b[3]; return __builtin_bit_cast(u32x4, h);
}
__device__ __forceinline__ bool wait_words(const GAS unsigned* word, unsigned need, unsigned* tmo, unsigned call) {
    const unsigned long long t0 = __builtin_amdgcn_s_memrealtime();
    for (unsigned n = 1;; ++n) {
        const unsigned v = __hip_atomic_load(word, __ATOMIC_RELAXED, __HIP_MEMORY_SCOPE_SYSTEM);
        if (__all((int)(v - need) >= 0)) return false;
        if ((n & 15u) == 0u) {
            if (__builtin_amdgcn_readfirstlane(__hip_atomic_load(tmo, __ATOMIC_RELAXED, __HIP_MEMORY_SCOPE_AGENT)) == call || __builtin_amdgcn_s_memrealtime() - t0 > DEADLINE) {
                __hip_atomic_store(tmo, call, __ATOMIC_RELAXED, __HIP_MEMORY_SCOPE_AGENT); return true; }
        }
        __builtin_amdgcn_s_sleep(1);
    }
}

__global__ __launch_bounds__(512, 2) __attribute__((amdgpu_num_vgpr(110))) void ag_gemm(Params p) {
    extern __shared__ __attribute__((aligned(16))) unsigned char shm[];
    LAS unsigned char* lds = (LAS unsigned char*)shm;
    const int tid = threadIdx.x, wid = __builtin_amdgcn_readfirstlane(tid >> 6), lane = tid & 63;
    const int b = blockIdx.x;
    char* myws = p.ws[p.rank];
    STAMP(0);
    const int wr = wid >> 2, wc = wid & 3;
    const int g = b & 7, idx = b >> 3;
    const int src = (p.rank + g) & (NRANK - 1);
    const int pm = 2 * src + (idx & 1), pn = idx >> 1;
    const int jp = (g + pn) & 7;
    const int mem = jp * 2 + (idx & 1);
    const char* wsrc = (const char*)(p.w + (size_t)(mem * 64 + (wid & 1) * 32) * N_ + pn * 256 + (wid >> 1) * 64);
    const unsigned wvoff = (unsigned)((lane >> 4) * 8 * N_ + (lane & 15) * 4) * 4u;
    char* bdst = myws + OFF_B16 + (size_t)((pn * 64 + mem) * 2) * HTB + ((wid >> 1) * 4 + (wid & 1)) * 1024;
    const unsigned lane16 = lane * 16;
    constexpr size_t WROUND = (size_t)1024 * N_ * 4, BROUND = (size_t)16 * 2 * HTB;
    unsigned* const tmo = (unsigned*)(myws + CB_TMO);
    unsigned long long bctr = (unsigned long long)myws + CB_BC + (pn * 32 + jp) * 128;
    asm volatile("" : "+s"(bctr));
#define WREGS "v224","v225","v226","v227","v228","v229","v230","v231","v232","v233","v234","v235","v236","v237","v238","v239","v240","v241","v242","v243","v244","v245","v246","v247","v248","v249","v250","v251","v252","v253","v254","v255"
#define LW1(R, r, j) asm volatile("global_load_dwordx4 " R ", %0, %1 nt" :: "v"(wvoff), "s"(wsrc + (size_t)(r) * WROUND + (size_t)(j) * N_ * 4) : "memory")
#define LOADW01(r) do { LW1("v[224:227]", r, 0); LW1("v[228:231]", r, 1); } while (0)
#define LOADW23(r) do { LW1("v[232:235]", r, 2); LW1("v[236:239]", r, 3); } while (0)
#define LOADW45(r) do { LW1("v[240:243]", r, 4); LW1("v[244:247]", r, 5); } while (0)
#define LOADW67(r) do { LW1("v[248:251]", r, 6); LW1("v[252:255]", r, 7); } while (0)
#define LOADW0() do { LOADW01(0); LOADW23(0); LOADW45(0); LOADW67(0); } while (0)
#define CV1(h, a0, a1, a2, a3, a4, a5, a6, a7) asm volatile("v_cvt_pk_f16_f32 %0, v" #a0 ", v" #a1 "\n\tv_cvt_pk_f16_f32 %1, v" #a2 ", v" #a3 "\n\tv_cvt_pk_f16_f32 %2, v" #a4 ", v" #a5 "\n\tv_cvt_pk_f16_f32 %3, v" #a6 ", v" #a7 \
        : "=v"(h[0]), "=v"(h[1]), "=v"(h[2]), "=v"(h[3]))
#define SCHED __builtin_amdgcn_sched_barrier(0)
#define STOREW01(r) do { const __amdgpu_buffer_rsrc_t _rs = __builtin_amdgcn_make_buffer_rsrc(bdst + (size_t)(r) * BROUND, 0, 4096, 0x00020000); u32x4 h0; u32x4 h1; \
        CV1(h0, 224, 228, 232, 236, 240, 244, 248, 252); CV1(h1, 225, 229, 233, 237, 241, 245, 249, 253); SCHED; \
        __builtin_amdgcn_raw_buffer_store_b128(h0, _rs, lane16, 0, 16); __builtin_amdgcn_raw_buffer_store_b128(h1, _rs, lane16 + 2048, 0, 16); SCHED; asm volatile("s_nop 1"); } while (0)
#define STOREW23(r) do { const __amdgpu_buffer_rsrc_t _rs = __builtin_amdgcn_make_buffer_rsrc(bdst + (size_t)(r) * BROUND + HTB, 0, 4096, 0x00020000); u32x4 h2; u32x4 h3; \
        CV1(h2, 226, 230, 234, 238, 242, 246, 250, 254); CV1(h3, 227, 231, 235, 239, 243, 247, 251, 255); SCHED; \
        __builtin_amdgcn_raw_buffer_store_b128(h2, _rs, lane16, 0, 16); __builtin_amdgcn_raw_buffer_store_b128(h3, _rs, lane16 + 2048, 0, 16); SCHED; asm volatile("s_nop 1"); } while (0)
#define SIGNALB(r) do { if (lane == 0) asm volatile("global_atomic_add %0, %1, %2 sc1" :: "v"(0u), "v"(1u), "s"(bctr + (r) * 1024) : "memory"); } while (0)
#define SIGNALA(n) do { if (lane < NRANK) asm volatile("global_atomic_add v[222:223], %0, off sc1" :: "v"(n) : "memory"); } while (0)
#define WAIT_V(n) asm volatile("s_waitcnt vmcnt(" #n ")" ::: "memory")
#define WAIT_L(n) asm volatile("s_waitcnt lgkmcnt(" #n ")" ::: "memory")
#define BAR __builtin_amdgcn_s_barrier()
    asm volatile("" ::: "v220", "v221", "v222", "v223", WREGS);

    const bool clsA = jp < 2, clsC = jp >= 5;
    const int xb = (7 - jp) * 32 + idx;
    const unsigned ldsw = __builtin_amdgcn_readfirstlane((unsigned)(size_t)lds + (unsigned)wid * 2048u);
    const unsigned pfoff = lane * 64;
    LAS unsigned* ctlw = (LAS unsigned*)(lds + 8 * HTB + 256);
    const char* fB = myws + CB_BC + pn * 32 * 128;
    const unsigned aflag4 = CB_AC + (src * 64 + (lane & 1) * 32 + ((lane >> 1) & 3)) * 128;
    const unsigned aflag64 = CB_AC + (src * 64 + lane) * 128;
    asm volatile("global_load_dword v220, %0, %1 sc1" :: "v"(0u), "s"(myws + CB_TK + b * 128) : "memory");
    unsigned call;
    const char* cA0;
#define CALL_FROM_V220() do { asm volatile("v_readfirstlane_b32 %0, v220" : "=s"(call)); call += 1u; cA0 = p.ws[src] + OFF_A16 + (size_t)(call & 1) * A16_BYTES; } while (0)
#define PFO(off) asm volatile("s_mov_b32 m0, %2\n\ts_nop 0\n\tglobal_load_lds_dword %0, %1" :: "v"(pfoff), "s"(cA0 + (off)), "s"(ldsw - wid * 2048 + 8 * HTB) : "memory")
#define PF(q) PFO(idx * 4096 + (size_t)(q) * 131072)
    if (wid < 4) {
        f32x4 xv[8];
        asm volatile("global_load_dwordx2 v[222:223], %0, %1" :: "v"((lane & 7) * 8), "s"((const char*)__builtin_amdgcn_kernarg_segment_ptr() + __builtin_offsetof(Params, ws)) : "memory");
        if (tid == 0) { ctlw[0] = 0u; ctlw[1] = 0u; ctlw[2] = 0u; WAIT_L(0); }
        if (!clsC) { if (!clsA) __builtin_amdgcn_s_sleep(32); LOADW0(); }
        { const char* s = (const char*)(p.x + (size_t)((xb & 3) * 128 + wid * 32) * K_ + (xb >> 2) * 64);
          const unsigned xo = (unsigned)((lane & 15) * K_ + (lane >> 4) * 8) * 4u;
          asm volatile("global_load_dwordx4 %0, %4, %5\n\tglobal_load_dwordx4 %1, %4, %5 offset:16\n\tglobal_load_dwordx4 %2, %4, %5 offset:128\n\tglobal_load_dwordx4 %3, %4, %5 offset:144"
                       : "=&v"(xv[0]), "=&v"(xv[1]), "=&v"(xv[2]), "=&v"(xv[3]) : "v"(xo), "s"(s) : "memory");
          asm volatile("global_load_dwordx4 %0, %4, %5\n\tglobal_load_dwordx4 %1, %4, %5 offset:16\n\tglobal_load_dwordx4 %2, %4, %5 offset:128\n\tglobal_load_dwordx4 %3, %4, %5 offset:144"
                       : "=&v"(xv[4]), "=&v"(xv[5]), "=&v"(xv[6]), "=&v"(xv[7]) : "v"(xo), "s"(s + (size_t)16 * K_ * 4) : "memory"); }
        BAR;
        if (!clsC) { WAIT_V(8); STOREW01(0); STOREW23(0); WAIT_V(4); } else WAIT_V(0);
        asm volatile("" : "+v"(xv[0]), "+v"(xv[1]), "+v"(xv[2]), "+v"(xv[3]), "+v"(xv[4]), "+v"(xv[5]), "+v"(xv[6]), "+v"(xv[7]) :: "memory");
        CALL_FROM_V220();
        asm volatile("v_lshl_add_u64 v[222:223], v[222:223], 0, %0" :: "s"((unsigned long long)(CB_AC + ((p.rank * 2 + ((xb >> 1) & 1)) * 32 + (xb >> 3)) * 128)));
        { char* dst = myws + OFF_A16 + (size_t)(call & 1) * A16_BYTES + (size_t)xb * HTB + wid * 4096;
          const __amdgpu_buffer_rsrc_t rs = __builtin_amdgcn_make_buffer_rsrc(dst, 0, 4096, 0x00020000);
#pragma unroll
          for (int i = 0; i < 4; ++i) __builtin_amdgcn_raw_buffer_store_b128(cvt8(xv[2 * i], xv[2 * i + 1]), rs, lane16 + i * 1024, 0, 17); }
        SCHED;
        if (jp <= 2) { WAIT_V(4); SIGNALB(0); }
        if (clsC) {
            WAIT_V(0);
            unsigned arr = 0;
            if (lane == 0) arr = __hip_atomic_fetch_add(ctlw + 2, 1u, __ATOMIC_RELAXED, __HIP_MEMORY_SCOPE_WORKGROUP);
            if (__builtin_amdgcn_readfirstlane(arr) == 3u) SIGNALA(4u);
        }
        STAMP(7);
        if (wid == 0) {
            const bool to = wait_words((const GAS unsigned*)(fB + (lane & 1) * 128), 16u * call, tmo, call);
            if (to && lane == 0) ctlw[0] = 1u;
            WAIT_L(0);
        }
    } else {
        if (!clsC) { if (!clsA) __builtin_amdgcn_s_sleep(32); LOADW0(); BAR; WAIT_V(0); STOREW01(0); STOREW23(0); if (jp <= 2) { WAIT_V(0); SIGNALB(0); } }
        else { BAR; WAIT_V(0); }
        CALL_FROM_V220();
        if (wid == 4 || jp == 5) {
            const bool to = wait_words((const GAS unsigned*)(myws + aflag4), 16u * call, tmo, call);
            if (to && lane == 0) ctlw[1] = 1u;
            WAIT_L(0);
            STAMPW(8, 4);
            if (jp == 5) {
                const unsigned u = (((pn >> 3) * 2 + (idx & 1)) * 4 + (wid - 4)) * 4096;
                PFO(u); WAIT_V(0); PFO(u + 65536); WAIT_V(0); PFO(u + 131072);
            }
        }
    }
    STAMP(1);
    BAR;
    asm volatile("" ::: "memory");
    if (tid == 0) __hip_atomic_store((unsigned*)(myws + CB_TK + b * 128), call, __ATOMIC_RELAXED, __HIP_MEMORY_SCOPE_AGENT);
    unsigned bad = 0;
    STAMP(6);
    const char* cA = cA0 + (pm & 1) * 2 * HTB;
    const char* cB = myws + OFF_B16 + (size_t)pn * 64 * 2 * HTB;
    const unsigned voff = wid * 2048 + lane * 16;
    const int aoff = wr * 8192 + lane * 16, boff = wc * 4096 + lane * 16;
    const unsigned fneed = 16u * call;
#define SA(b, h) (((b) * 2 + (h)) * HTB)
#define SB(b, h) ((4 + (b) * 2 + (h)) * HTB)
#define STAGE(bufoff, gbase, AUX) asm volatile("s_add_u32 m0, %1, %3\n\ts_nop 0\n\tglobal_load_lds_dwordx4 %0, %2" AUX "\n\tglobal_load_lds_dwordx4 %0, %2 offset:1024" AUX \
        :: "v"(voff), "s"(ldsw), "s"((const char*)(gbase)), "i"(bufoff) : "memory", "scc")
#define STA(b, h, ptr) STAGE(SA(b, h), ptr, "")
#define STB(b, h, ptr) STAGE(SB(b, h), ptr, " sc1")
#define LDA(dst, b, h) do { _Pragma("unroll") for (int m = 0; m < 4; ++m) _Pragma("unroll") for (int k = 0; k < 2; ++k) dst[m][k] = *(const LAS half8*)(lds + SA(b, h) + aoff + m * 2048 + k * 1024); } while (0)
#define LDB(dst, b, h) do { _Pragma("unroll") for (int n = 0; n < 2; ++n) _Pragma("unroll") for (int k = 0; k < 2; ++k) dst[n][k] = *(const LAS half8*)(lds + SB(b, h) + boff + n * 2048 + k * 1024); } while (0)
#define MMA(ai, bj, At, Bt) do { __builtin_amdgcn_s_setprio(1); _Pragma("unroll") for (int m = 0; m < 4; ++m) _Pragma("unroll") for (int nn = 0; nn < 2; ++nn) { \
        const int n = (m & 1) ? 1 - nn : nn, k = nn & 1; \
        asm volatile("v_mfma_f32_16x16x32_f16 %0, %1, %2, %0\n\tv_mfma_f32_16x16x32_f16 %0, %3, %4, %0" : "+v"(acc[ai][bj][m][n]) : "v"(At[m][k]), "v"(Bt[n][k]), "v"(At[m][1 - k]), "v"(Bt[n][1 - k])); } __builtin_amdgcn_s_setprio(0); } while (0)
#define FLAGLD(q) asm volatile("global_load_dword v220, %0, %1 sc0 sc1" :: "v"(0u), "s"(fB + (q) * 128) : "memory")
#define FLAGCK(q) do { unsigned fv; asm volatile("v_readfirstlane_b32 %0, v220" : "=s"(fv)); if ((int)(fv - fneed) < 0) { if (wait_words((const GAS unsigned*)(fB + (q) * 128), fneed, tmo, call)) bad = 1u; \
        WAIT_V(0); } } while (0)
    f32x4 acc[2][2][4][2];
#pragma unroll
    for (int a = 0; a < 2; ++a)
#pragma unroll
        for (int bb = 0; bb < 2; ++bb)
#pragma unroll
            for (int m = 0; m < 4; ++m)
#pragma unroll
                for (int n = 0; n < 2; ++n) { acc[a][bb][m][n] = (f32x4){0.f, 0.f, 0.f, 0.f}; asm volatile("" : "+v"(acc[a][bb][m][n])); }
    half8 At[4][2], B0[2][2], B1[2][2];
    STB(0, 0, cB); STA(0, 0, cA); STB(0, 1, cB + HTB); STA(0, 1, cA + HTB);
    if (wr == 1) BAR;
    WAIT_V(4); BAR;
    STB(1, 0, cB + 32768); STA(1, 0, cA + 65536); STB(1, 1, cB + 32768 + HTB);
    WAIT_V(6); BAR;
    const int pstart = jp + (wid & 1);
    unsigned mLd = 0x010101u << pstart;
    if (clsC) mLd |= 1u << (jp == 5 ? 0 : (wid & 1));
    unsigned mSg = mLd << 2, mLS = mLd | (mLd << 1);
    if (jp == 3 || jp == 4) mSg |= 1u;
    unsigned mFl = wr == 0 ? 0x11111111u << wc : 0u;
    unsigned mPf = wr == 1 ? 0x11111111u << wc : 0u;
    unsigned mEv = wid == 4 ? 0x202u : (wid < 4 && !clsC) ? 1u : 0u;
    unsigned mAs = wid == 4 ? 0x101u : 0u;
#define RFL(x) x = __builtin_amdgcn_readfirstlane(x)
    RFL(mLd); RFL(mSg); RFL(mFl); RFL(mPf); RFL(mEv); RFL(mAs); RFL(mLS);
    asm volatile("" : "+s"(mLd), "+s"(mSg), "+s"(mFl), "+s"(mPf), "+s"(mEv), "+s"(mAs), "+s"(mLS));
#define BIT(m) UNL(((m) >> i) & 1u)
#define RND(off) (i - (off) < pstart ? 0 : ((i - (off) - pstart) >> 3) + 1)
#define Q2 (i + 2 < 32 ? i + 2 : 31)
    const char* pa = cA + 131072;
    const char* pb = cB + 65536;
    for (int i = 0; i < NT / 2 - 1; ++i) {
        SCHED;
        LDB(B0, 0, 0); SCHED; LDA(At, 0, 0); STA(1, 1, pa - 65536 + HTB);
        WAIT_L(8); BAR; WAIT_L(0); MMA(0, 0, At, B0); BAR; SCHED;
        LDB(B1, 0, 1); STB(0, 0, pb); if (BIT(mLd)) LOADW01(RND(0));
        BAR; WAIT_L(0); MMA(0, 1, At, B1); BAR; SCHED;
        LDA(At, 0, 1); STA(0, 0, pa); if (BIT(mLd)) LOADW23(RND(0));
        BAR; WAIT_L(0); MMA(1, 0, At, B0); BAR; SCHED;
        STB(0, 1, pb + HTB);
        if (BIT(mFl)) { FLAGLD(Q2); if (BIT(mLd)) WAIT_V(11); else WAIT_V(7); }
        else if (BIT(mLd)) WAIT_V(10); else WAIT_V(6);
        if (BIT(mSg)) SIGNALB(RND(2));
        if (BIT(mEv)) {
            if (wid < 4) SIGNALA(1u);
            else {
                const unsigned need = (i == 1 && (lane & 31) >= 12) ? 0u : fneed;
                unsigned av; asm volatile("v_mov_b32 %0, v221" : "=v"(av));
                if (!__all((int)(av - need) >= 0)) { if (wait_words((const GAS unsigned*)(myws + aflag64), need, tmo, call)) { if (lane == 0) ctlw[1] = 1u; } WAIT_V(0); }
            }
        }
        BAR; MMA(1, 1, At, B1); BAR; SCHED;
        LDB(B0, 1, 0); SCHED; LDA(At, 1, 0); STA(0, 1, pa + HTB);
        WAIT_L(8); BAR; WAIT_L(0); MMA(0, 0, At, B0); BAR; SCHED;
        LDB(B1, 1, 1); STB(1, 0, pb + 32768);
        if (BIT(mPf)) PF(Q2);
        if (BIT(mLS)) { if (BIT(mLd)) LOADW45(RND(0)); else STOREW01(RND(1)); }
        BAR; WAIT_L(0); MMA(0, 1, At, B1); BAR; SCHED;
        LDA(At, 1, 1); STA(1, 0, pa + 65536);
        if (BIT(mLS)) { if (BIT(mLd)) LOADW67(RND(0)); else STOREW23(RND(1)); }
        BAR; WAIT_L(0); MMA(1, 0, At, B0); BAR; SCHED;
        STB(1, 1, pb + 32768 + HTB);
        pa += 131072; pb += 65536;
        if (BIT(mPf)) { if (BIT(mLS)) WAIT_V(11); else WAIT_V(7); }
        else { if (BIT(mLS)) WAIT_V(10); else WAIT_V(6); if (BIT(mFl)) FLAGCK(Q2); }
        if (BIT(mAs)) asm volatile("global_load_dword v221, %0, %1 sc0 sc1" :: "v"(aflag64), "s"(myws) : "memory");
        BAR; MMA(1, 1, At, B1); BAR; SCHED;
    }
    {
        half8 Aa[2][2], Ab[2][2];
#define LDA1(dst, kt, ai, m) do { _Pragma("unroll") for (int k = 0; k < 2; ++k) dst[kt][k] = *(const LAS half8*)(lds + SA(kt, ai) + aoff + (m) * 2048 + k * 1024); } while (0)
#define MF2(ai, bj, m, n, A, B) asm volatile("v_mfma_f32_16x16x32_f16 %0, %1, %2, %0\n\tv_mfma_f32_16x16x32_f16 %0, %3, %4, %0" : "+v"(acc[ai][bj][m][n]) : "v"(A[0]), "v"(B[n][0]), "v"(A[1]), "v"(B[n][1]))
#define MF2R(ai, bj, m, n, A, R0, R1) asm volatile("v_mfma_f32_16x16x32_f16 %0, %1, " R0 ", %0\n\tv_mfma_f32_16x16x32_f16 %0, %2, " R1 ", %0" : "+v"(acc[ai][bj][m][n]) : "v"(A[0]), "v"(A[1]))
#define MF4(ai, bj, m, n, A, B, R0, R1) asm volatile("v_mfma_f32_16x16x32_f16 %0, %1, %2, %0\n\tv_mfma_f32_16x16x32_f16 %0, %3, %4, %0\n\tv_mfma_f32_16x16x32_f16 %0, %5, " R0 ", %0\n\tv_mfma_f32_16x16x32_f16 %0, %6, " R1 ", %0" \
        : "+v"(acc[ai][bj][m][n]) : "v"(A[0][0]), "v"(B[n][0]), "v"(A[0][1]), "v"(B[n][1]), "v"(A[1][0]), "v"(A[1][1]))
#define MMB(ai, m, A) do { __builtin_amdgcn_s_setprio(1); MF4(ai, 0, m, 0, A, B0, "v[224:227]", "v[228:231]"); MF4(ai, 0, m, 1, A, B0, "v[232:235]", "v[236:239]"); \
        MF4(ai, 1, m, 0, A, B1, "v[240:243]", "v[244:247]"); MF4(ai, 1, m, 1, A, B1, "v[248:251]", "v[252:255]"); __builtin_amdgcn_s_setprio(0); } while (0)
        LDA1(Aa, 0, 0, 0); LDB(B0, 0, 0); LDB(B1, 0, 1); STA(1, 1, pa - 65536 + HTB);
        if (UNL(bad) && lane == 0) ctlw[0] = 1u;
        if (wr == 0) BAR;
        WAIT_L(0); MF2(0, 0, 0, 0, Aa[0], B0); MF2(0, 0, 0, 1, Aa[0], B0); MF2(0, 1, 0, 0, Aa[0], B1); MF2(0, 1, 0, 1, Aa[0], B1); SCHED;
        WAIT_V(2); BAR; SCHED;
        asm volatile("ds_read_b128 v[224:227], %0\n\tds_read_b128 v[228:231], %0 offset:1024\n\tds_read_b128 v[232:235], %0 offset:2048\n\tds_read_b128 v[236:239], %0 offset:3072\n\t"
                     "ds_read_b128 v[240:243], %0 offset:16384\n\tds_read_b128 v[244:247], %0 offset:17408\n\tds_read_b128 v[248:251], %0 offset:18432\n\tds_read_b128 v[252:255], %0 offset:19456"
                     :: "v"((unsigned)(size_t)lds + SB(1, 0) + boff) : "memory");
        LDA1(Aa, 1, 0, 0);
        bad = *(volatile LAS unsigned*)ctlw | *(volatile LAS unsigned*)(ctlw + 1);
        WAIT_L(0);
        MF2R(0, 0, 0, 0, Aa[1], "v[224:227]", "v[228:231]"); MF2R(0, 0, 0, 1, Aa[1], "v[232:235]", "v[236:239]");
        MF2R(0, 1, 0, 0, Aa[1], "v[240:243]", "v[244:247]"); MF2R(0, 1, 0, 1, Aa[1], "v[248:251]", "v[252:255]"); SCHED;
        LDA1(Aa, 0, 0, 1); LDA1(Aa, 1, 0, 1);
        bad = __builtin_amdgcn_readfirstlane(bad);
        int tz = threadIdx.x; asm volatile("" : "+v"(tz));
        const unsigned ovoff = (unsigned)(((tz >> 4) & 3) * 4 * N_ + (tz & 15) * 4) * 4u;
        const __amdgpu_buffer_rsrc_t ors = __builtin_amdgcn_make_buffer_rsrc(p.out + (size_t)(pm * 256 + wr * 64) * N_ + pn * 256 + wc * 64, 0, 192 * N_ * 4, 0x00020000);
#define EPI(ai, m) do { const float k1 = -2.0f * 0.7978845608028654f * 1.4426950408889634f, k2 = k1 * 0.044715f; f32x4 v[4]; \
            _Pragma("unroll") for (int t = 0; t < 4; ++t) _Pragma("unroll") for (int h = 0; h < 2; ++h) { \
                const f32x2 y = {acc[ai][t >> 1][m][t & 1][2 * h], acc[ai][t >> 1][m][t & 1][2 * h + 1]}; \
                const f32x2 z = y * __builtin_elementwise_fma(y * y, (f32x2){k2, k2}, (f32x2){k1, k1}); \
                f32x2 e = {__builtin_amdgcn_exp2f(z[0]), __builtin_amdgcn_exp2f(z[1])}; e += 1.0f; \
                float o0 = y[0] * __builtin_amdgcn_rcpf(e[0]); float o1 = y[1] * __builtin_amdgcn_rcpf(e[1]); \
                asm("" : "+v"(o0)); asm("" : "+v"(o1)); v[2 * h][t] = o0; v[2 * h + 1][t] = o1; } \
            _Pragma("unroll") for (int j = 0; j < 4; ++j) __builtin_amdgcn_raw_buffer_store_b128(__builtin_bit_cast(u32x4, v[j]), ors, ovoff, ((ai) * 128 + (m) * 16 + j) * N_ * 4, 2); } while (0)
        WAIT_V(0); BAR; SCHED;
        if (UNL(bad)) {
            const unsigned q = 0x7fc00000u;
#pragma unroll 1
            for (int i = 0; i < 32; ++i) __builtin_amdgcn_raw_buffer_store_b128((u32x4){q, q, q, q}, ors, ovoff, ((i >> 4) * 128 + ((i >> 2) & 3) * 16 + (i & 3)) * N_ * 4, 0);
        } else {
            STAMP(3);
            WAIT_L(0); MMB(0, 1, Aa); SCHED;
            EPI(0, 0); SCHED; LDA1(Aa, 0, 0, 2); LDA1(Aa, 1, 0, 2); LDA1(Ab, 0, 0, 3); LDA1(Ab, 1, 0, 3); SCHED;
            WAIT_L(4); MMB(0, 2, Aa); SCHED;
            LDA1(Aa, 0, 1, 0); LDA1(Aa, 1, 1, 0); SCHED; EPI(0, 1); SCHED;
            MMB(0, 3, Ab); SCHED;
            LDA1(Ab, 0, 1, 1); LDA1(Ab, 1, 1, 1); SCHED; EPI(0, 2); SCHED;
            MMB(1, 0, Aa); SCHED;
            LDA1(Aa, 0, 1, 2); LDA1(Aa, 1, 1, 2); SCHED; EPI(0, 3); SCHED;
            MMB(1, 1, Ab); SCHED;
            LDA1(Ab, 0, 1, 3); LDA1(Ab, 1, 1, 3); SCHED; EPI(1, 0); SCHED;
            MMB(1, 2, Aa); SCHED;
            EPI(1, 1); SCHED;
            MMB(1, 3, Ab); SCHED;
            EPI(1, 2); SCHED;
            asm volatile("s_nop 7" ::: "memory"); SCHED;
            EPI(1, 3);
        }
    }
    STAMP(4);
    WAIT_V(0);
    STAMP(5);
}

extern "C" void kernel_launch(int rank, int nranks, ncclComm_t comm, hipStream_t stream,
                              void* const* d_in, const int* in_sizes, int n_in, void* d_out, int out_size,
                              const int* in_dtypes, int out_dtype, void* const* d_out_all, void* const* d_ws_all, size_t ws_bytes) {
    static int ok = -1;
    if (ok < 0) {
        int dev = 0, cus = 0;
        ok = hipGetDevice(&dev) == hipSuccess && hipDeviceGetAttribute(&cus, hipDeviceAttributeMultiprocessorCount, dev) == hipSuccess && cus >= 256 &&
             hipFuncSetAttribute((const void*)ag_gemm, hipFuncAttributeMaxDynamicSharedMemorySize, LDS_BYTES) == hipSuccess;
    }
    if (!ok || nranks != NRANK || (unsigned)rank >= (unsigned)NRANK || ws_bytes < WS_NEED || n_in != 2 || in_sizes[0] != MS * K_ || in_sizes[1] != K_ * N_ || out_size != M_ * N_) {
        fprintf(stderr, "ag_gemm: unsupported configuration\n");
        return;
    }
    Params p; p.x = (const float*)d_in[0]; p.w = (const float*)d_in[1]; p.out = (float*)d_out; p.rank = rank;
    for (int q = 0; q < NRANK; ++q) p.ws[q] = (char*)d_ws_all[q];
    ag_gemm<<<256, 512, LDS_BYTES, stream>>>(p);
}
```

```cpp
#include <hip/hip_runtime.h>
#include <rccl/rccl.h>
#include <cstdio>

#define UNL(x) __builtin_expect(!!(x), 0)
#define LAS __attribute__((address_space(3)))
#define GAS __attribute__((address_space(1)))
typedef _Float16 half8 __attribute__((ext_vector_type(8)));
typedef float f32x4 __attribute__((ext_vector_type(4)));
typedef float f32x2 __attribute__((ext_vector_type(2)));
typedef unsigned u32x4 __attribute__((ext_vector_type(4)));

constexpr int NRANK = 8;
constexpr int M_ = 4096, K_ = 4096, N_ = 4096, MS = M_ / NRANK;
constexpr int NT = K_ / 64;
constexpr int HTB = 16384;
constexpr int LDS_BYTES = 8 * HTB + 512;
constexpr size_t OFF_A16 = 1u << 20;
constexpr size_t A16_BYTES = (size_t)MS * K_ * 2;
constexpr size_t OFF_B16 = 16u << 20;
constexpr size_t WS_NEED = OFF_B16 + (size_t)K_ * N_ * 2;
constexpr unsigned long long DEADLINE = 50000000ull;
constexpr unsigned CB_TMO = 0;
constexpr unsigned CB_TK = 4096;
constexpr unsigned CB_BC = CB_TK + 256 * 128;
constexpr unsigned CB_AC = CB_BC + 512 * 128;
static_assert(CB_AC + 512 * 128 <= OFF_A16 && OFF_A16 + 2 * A16_BYTES <= OFF_B16, "workspace layout");

#define STAMPW(i, w) do {} while (0)
#define STAMP(i) STAMPW(i, 0)

struct Params { const float* x; const float* w; float* out; char* ws[NRANK]; int rank; };

__device__ __forceinline__ u32x4 cvt8(f32x4 a, f32x4 b) {
    half8 h; h[0] = (_Float16)a[0]; h[1] = (_Float16)a[1]; h[2] = (_Float16)a[2]; h[3] = (_Float16)a[3];
    h[4] = (_Float16)b[0]; h[5] = (_Float16)b[1]; h[6] = (_Float16)b[2]; h[7] = (_Float16)b[3]; return __builtin_bit_cast(u32x4, h);
}
__device__ __forceinline__ bool wait_words(const GAS unsigned* word, unsigned need, unsigned* tmo, unsigned call) {
    const unsigned long long t0 = __builtin_amdgcn_s_memrealtime();
    for (unsigned n = 1;; ++n) {
        const unsigned v = __hip_atomic_load(word, __ATOMIC_RELAXED, __HIP_MEMORY_SCOPE_SYSTEM);
        if (__all((int)(v - need) >= 0)) return false;
        if ((n & 15u) == 0u) {
            if (__builtin_amdgcn_readfirstlane(__hip_atomic_load(tmo, __ATOMIC_RELAXED, __HIP_MEMORY_SCOPE_AGENT)) == call || __builtin_amdgcn_s_memrealtime() - t0 > DEADLINE) {
                __hip_atomic_store(tmo, call, __ATOMIC_RELAXED, __HIP_MEMORY_SCOPE_AGENT); return true; }
        }
        __builtin_amdgcn_s_sleep(1);
    }
}

__global__ __launch_bounds__(512, 2) __attribute__((amdgpu_num_vgpr(110))) void ag_gemm(Params p) {
    extern __shared__ __attribute__((aligned(16))) unsigned char shm[];
    LAS unsigned char* lds = (LAS unsigned char*)shm;
    const int tid = threadIdx.x, wid = __builtin_amdgcn_readfirstlane(tid >> 6), lane = tid & 63;
    const int b = blockIdx.x;
    char* myws = p.ws[p.rank];
    STAMP(0);
    const int wr = wid >> 2, wc = wid & 3;
    const int g = b & 7, idx = b >> 3;
    const int src = (p.rank + g) & (NRANK - 1);
    const int pm = 2 * src + (idx & 1), pn = idx >> 1;
    const int jp = (g + pn) & 7;
    const int mem = jp * 2 + (idx & 1);
    const char* wsrc = (const char*)(p.w + (size_t)(mem * 64 + (wid & 1) * 32) * N_ + pn * 256 + (wid >> 1) * 64);
    const unsigned wvoff = (unsigned)((lane >> 4) * 8 * N_ + (lane & 15) * 4) * 4u;
    char* bdst = myws + OFF_B16 + (size_t)((pn * 64 + mem) * 2) * HTB + ((wid >> 1) * 4 + (wid & 1)) * 1024;
    const unsigned lane16 = lane * 16;
    constexpr size_t WROUND = (size_t)1024 * N_ * 4, BROUND = (size_t)16 * 2 * HTB;
    unsigned* const tmo = (unsigned*)(myws + CB_TMO);
    unsigned long long bctr = (unsigned long long)myws + CB_BC + (pn * 32 + jp) * 128;
    asm volatile("" : "+s"(bctr));
#define WREGS "v224","v225","v226","v227","v228","v229","v230","v231","v232","v233","v234","v235","v236","v237","v238","v239","v240","v241","v242","v243","v244","v245","v246","v247","v248","v249","v250","v251","v252","v253","v254","v255"
#define LW1(R, r, j) asm volatile("global_load_dwordx4 " R ", %0, %1 nt" :: "v"(wvoff), "s"(wsrc + (size_t)(r) * WROUND + (size_t)(j) * N_ * 4) : "memory")
#define LOADW01(r) do { LW1("v[224:227]", r, 0); LW1("v[228:231]", r, 1); } while (0)
#define LOADW23(r) do { LW1("v[232:235]", r, 2); LW1("v[236:239]", r, 3); } while (0)
#define LOADW45(r) do { LW1("v[240:243]", r, 4); LW1("v[244:247]", r, 5); } while (0)
#define LOADW67(r) do { LW1("v[248:251]", r, 6); LW1("v[252:255]", r, 7); } while (0)
#define LOADW0() do { LOADW01(0); LOADW23(0); LOADW45(0); LOADW67(0); } while (0)
#define CV1(h, a0, a1, a2, a3, a4, a5, a6, a7) asm volatile("v_cvt_pk_f16_f32 %0, v" #a0 ", v" #a1 "\n\tv_cvt_pk_f16_f32 %1, v" #a2 ", v" #a3 "\n\tv_cvt_pk_f16_f32 %2, v" #a4 ", v" #a5 "\n\tv_cvt_pk_f16_f32 %3, v" #a6 ", v" #a7 \
        : "=v"(h[0]), "=v"(h[1]), "=v"(h[2]), "=v"(h[3]))
#define SCHED __builtin_amdgcn_sched_barrier(0)
#define STOREW01(r) do { const __amdgpu_buffer_rsrc_t _rs = __builtin_amdgcn_make_buffer_rsrc(bdst + (size_t)(r) * BROUND, 0, 4096, 0x00020000); u32x4 h0; u32x4 h1; \
        CV1(h0, 224, 228, 232, 236, 240, 244, 248, 252); CV1(h1, 225, 229, 233, 237, 241, 245, 249, 253); SCHED; \
        __builtin_amdgcn_raw_buffer_store_b128(h0, _rs, lane16, 0, 16); __builtin_amdgcn_raw_buffer_store_b128(h1, _rs, lane16 + 2048, 0, 16); SCHED; asm volatile("s_nop 1"); } while (0)
#define STOREW23(r) do { const __amdgpu_buffer_rsrc_t _rs = __builtin_amdgcn_make_buffer_rsrc(bdst + (size_t)(r) * BROUND + HTB, 0, 4096, 0x00020000); u32x4 h2; u32x4 h3; \
        CV1(h2, 226, 230, 234, 238, 242, 246, 250, 254); CV1(h3, 227, 231, 235, 239, 243, 247, 251, 255); SCHED; \
        __builtin_amdgcn_raw_buffer_store_b128(h2, _rs, lane16, 0, 16); __builtin_amdgcn_raw_buffer_store_b128(h3, _rs, lane16 + 2048, 0, 16); SCHED; asm volatile("s_nop 1"); } while (0)
#define SIGNALB(r) do { if (lane == 0) asm volatile("global_atomic_add %0, %1, %2 sc1" :: "v"(0u), "v"(1u), "s"(bctr + (r) * 1024) : "memory"); } while (0)
#define SIGNALA(n) do { if (lane < NRANK) asm volatile("global_atomic_add v[222:223], %0, off sc1" :: "v"(n) : "memory"); } while (0)
#define WAIT_V(n) asm volatile("s_waitcnt vmcnt(" #n ")" ::: "memory")
#define WAIT_L(n) asm volatile("s_waitcnt lgkmcnt(" #n ")" ::: "memory")
#define BAR __builtin_amdgcn_s_barrier()
    asm volatile("" ::: "v220", "v221", "v222", "v223", WREGS);

    const bool clsA = jp < 2, clsC = jp >= 5;
    const int xb = (7 - jp) * 32 + idx;
    const unsigned ldsw = __builtin_amdgcn_readfirstlane((unsigned)(size_t)lds + (unsigned)wid * 2048u);
    const unsigned pfoff = lane * 64;
    LAS unsigned* ctlw = (LAS unsigned*)(lds + 8 * HTB + 256);
    const char* fB = myws + CB_BC + pn * 32 * 128;
    const unsigned aflag4 = CB_AC + (src * 64 + (lane & 1) * 32 + ((lane >> 1) & 3)) * 128;
    const unsigned aflag64 = CB_AC + (src * 64 + lane) * 128;
    asm volatile("global_load_dword v220, %0, %1 sc1" :: "v"(0u), "s"(myws + CB_TK + b * 128) : "memory");
    unsigned call;
    const char* cA0;
#define CALL_FROM_V220() do { asm volatile("v_readfirstlane_b32 %0, v220" : "=s"(call)); call += 1u; cA0 = p.ws[src] + OFF_A16 + (size_t)(call & 1) * A16_BYTES; } while (0)
#define PFO(off) asm volatile("s_mov_b32 m0, %2\n\ts_nop 0\n\tglobal_load_lds_dword %0, %1" :: "v"(pfoff), "s"(cA0 + (off)), "s"(ldsw - wid * 2048 + 8 * HTB) : "memory")
#define PF(q) PFO(idx * 4096 + (size_t)(q) * 131072)
    if (wid < 4) {
        f32x4 xv[8];
        asm volatile("global_load_dwordx2 v[222:223], %0, %1" :: "v"((lane & 7) * 8), "s"((const char*)__builtin_amdgcn_kernarg_segment_ptr() + __builtin_offsetof(Params, ws)) : "memory");
        if (tid == 0) { ctlw[0] = 0u; ctlw[1] = 0u; ctlw[2] = 0u; WAIT_L(0); }
        if (!clsC) { if (!clsA) __builtin_amdgcn_s_sleep(32); LOADW0(); }
        { const char* s = (const char*)(p.x + (size_t)((xb & 3) * 128 + wid * 32) * K_ + (xb >> 2) * 64);
          const unsigned xo = (unsigned)((lane & 15) * K_ + (lane >> 4) * 8) * 4u;
          asm volatile("global_load_dwordx4 %0, %4, %5\n\tglobal_load_dwordx4 %1, %4, %5 offset:16\n\tglobal_load_dwordx4 %2, %4, %5 offset:128\n\tglobal_load_dwordx4 %3, %4, %5 offset:144"
                       : "=&v"(xv[0]), "=&v"(xv[1]), "=&v"(xv[2]), "=&v"(xv[3]) : "v"(xo), "s"(s) : "memory");
          asm volatile("global_load_dwordx4 %0, %4, %5\n\tglobal_load_dwordx4 %1, %4, %5 offset:16\n\tglobal_load_dwordx4 %2, %4, %5 offset:128\n\tglobal_load_dwordx4 %3, %4, %5 offset:144"
                       : "=&v"(xv[4]), "=&v"(xv[5]), "=&v"(xv[6]), "=&v"(xv[7]) : "v"(xo), "s"(s + (size_t)16 * K_ * 4) : "memory"); }
        BAR;
        if (!clsC) { WAIT_V(8); STOREW01(0); STOREW23(0); WAIT_V(4); } else WAIT_V(0);
        asm volatile("" : "+v"(xv[0]), "+v"(xv[1]), "+v"(xv[2]), "+v"(xv[3]), "+v"(xv[4]), "+v"(xv[5]), "+v"(xv[6]), "+v"(xv[7]) :: "memory");
        CALL_FROM_V220();
        asm volatile("v_lshl_add_u64 v[222:223], v[222:223], 0, %0" :: "s"((unsigned long long)(CB_AC + ((p.rank * 2 + ((xb >> 1) & 1)) * 32 + (xb >> 3)) * 128)));
        { char* dst = myws + OFF_A16 + (size_t)(call & 1) * A16_BYTES + (size_t)xb * HTB + wid * 4096;
          const __amdgpu_buffer_rsrc_t rs = __builtin_amdgcn_make_buffer_rsrc(dst, 0, 4096, 0x00020000);
#pragma unroll
          for (int i = 0; i < 4; ++i) __builtin_amdgcn_raw_buffer_store_b128(cvt8(xv[2 * i], xv[2 * i + 1]), rs, lane16 + i * 1024, 0, 17); }
        SCHED;
        if (jp <= 2) { WAIT_V(4); SIGNALB(0); }
        if (clsC) {
            WAIT_V(0);
            unsigned arr = 0;
            if (lane == 0) arr = __hip_atomic_fetch_add(ctlw + 2, 1u, __ATOMIC_RELAXED, __HIP_MEMORY_SCOPE_WORKGROUP);
            if (__builtin_amdgcn_readfirstlane(arr) == 3u) SIGNALA(4u);
        }
        STAMP(7);
        if (wid == 0) {
            const bool to = wait_words((const GAS unsigned*)(fB + (lane & 1) * 128), 16u * call, tmo, call);
            if (to && lane == 0) ctlw[0] = 1u;
            WAIT_L(0);
        }
    } else {
        if (!clsC) { if (!clsA) __builtin_amdgcn_s_sleep(32); LOADW0(); BAR; WAIT_V(0); STOREW01(0); STOREW23(0); if (jp <= 2) { WAIT_V(0); SIGNALB(0); } }
        else { BAR; WAIT_V(0); }
        CALL_FROM_V220();
        if (wid == 4 || jp == 5) {
            const bool to = wait_words((const GAS unsigned*)(myws + aflag4), 16u * call, tmo, call);
            if (to && lane == 0) ctlw[1] = 1u;
            WAIT_L(0);
            STAMPW(8, 4);
            if (jp == 5) {
                const unsigned u = (((pn >> 3) * 2 + (idx & 1)) * 4 + (wid - 4)) * 4096;
                PFO(u); WAIT_V(0); PFO(u + 65536); WAIT_V(0); PFO(u + 131072);
            }
        }
    }
    STAMP(1);
    BAR;
    asm volatile("" ::: "memory");
    if (tid == 0) __hip_atomic_store((unsigned*)(myws + CB_TK + b * 128), call, __ATOMIC_RELAXED, __HIP_MEMORY_SCOPE_AGENT);
    unsigned bad = 0;
    STAMP(6);
    const char* cA = cA0 + (pm & 1) * 2 * HTB;
    const char* cB = myws + OFF_B16 + (size_t)pn * 64 * 2 * HTB;
    const unsigned voff = wid * 2048 + lane * 16;
    const int aoff = wr * 8192 + lane * 16, boff = wc * 4096 + lane * 16;
    const unsigned fneed = 16u * call;
#define SA(b, h) (((b) * 2 + (h)) * HTB)
#define SB(b, h) ((4 + (b) * 2 + (h)) * HTB)
#define STAGE(bufoff, gbase, AUX) asm volatile("s_add_u32 m0, %1, %3\n\ts_nop 0\n\tglobal_load_lds_dwordx4 %0, %2" AUX "\n\tglobal_load_lds_dwordx4 %0, %2 offset:1024" AUX \
        :: "v"(voff), "s"(ldsw), "s"((const char*)(gbase)), "i"(bufoff) : "memory", "scc")
#define STA(b, h, ptr) STAGE(SA(b, h), ptr, "")
#define STB(b, h, ptr) STAGE(SB(b, h), ptr, " sc1")
#define LDA(dst, b, h) do { _Pragma("unroll") for (int m = 0; m < 4; ++m) _Pragma("unroll") for (int k = 0; k < 2; ++k) dst[m][k] = *(const LAS half8*)(lds + SA(b, h) + aoff + m * 2048 + k * 1024); } while (0)
#define LDB(dst, b, h) do { _Pragma("unroll") for (int n = 0; n < 2; ++n) _Pragma("unroll") for (int k = 0; k < 2; ++k) dst[n][k] = *(const LAS half8*)(lds + SB(b, h) + boff + n * 2048 + k * 1024); } while (0)
#define MMA(ai, bj, At, Bt) do { _Pragma("unroll") for (int m = 0; m < 4; ++m) _Pragma("unroll") for (int nn = 0; nn < 2; ++nn) { \
        const int n = (m & 1) ? 1 - nn : nn, k = nn & 1; \
        asm volatile("v_mfma_f32_16x16x32_f16 %0, %1, %2, %0\n\tv_mfma_f32_16x16x32_f16 %0, %3, %4, %0" : "+v"(acc[ai][bj][m][n]) : "v"(At[m][k]), "v"(Bt[n][k]), "v"(At[m][1 - k]), "v"(Bt[n][1 - k])); } } while (0)
#define FLAGLD(q) asm volatile("global_load_dword v220, %0, %1 sc0 sc1" :: "v"(0u), "s"(fB + (q) * 128) : "memory")
#define FLAGCK(q) do { unsigned fv; asm volatile("v_readfirstlane_b32 %0, v220" : "=s"(fv)); if ((int)(fv - fneed) < 0) { if (wait_words((const GAS unsigned*)(fB + (q) * 128), fneed, tmo, call)) bad = 1u; \
        WAIT_V(0); } } while (0)
    f32x4 acc[2][2][4][2];
#pragma unroll
    for (int a = 0; a < 2; ++a)
#pragma unroll
        for (int bb = 0; bb < 2; ++bb)
#pragma unroll
            for (int m = 0; m < 4; ++m)
#pragma unroll
                for (int n = 0; n < 2; ++n) { acc[a][bb][m][n] = (f32x4){0.f, 0.f, 0.f, 0.f}; asm volatile("" : "+v"(acc[a][bb][m][n])); }
    half8 At[4][2], B0[2][2], B1[2][2];
    STB(0, 0, cB); STA(0, 0, cA); STB(0, 1, cB + HTB); STA(0, 1, cA + HTB);
    if (wr == 1) BAR;
    WAIT_V(4); BAR;
    STB(1, 0, cB + 32768); STA(1, 0, cA + 65536); STB(1, 1, cB + 32768 + HTB);
    WAIT_V(6); BAR;
    const int pstart = jp + (wid & 1);
    unsigned mLd = 0x010101u << pstart;
    if (clsC) mLd |= 1u << (jp == 5 ? 0 : (wid & 1));
    unsigned mSg = mLd << 2, mLS = mLd | (mLd << 1);
    if (jp == 3 || jp == 4) mSg |= 1u;
    unsigned mFl = wr == 0 ? 0x11111111u << wc : 0u;
    unsigned mPf = wr == 1 ? 0x11111111u << wc : 0u;
    unsigned mEv = wid == 4 ? 0x202u : (wid < 4 && !clsC) ? 1u : 0u;
    unsigned mAs = wid == 4 ? 0x101u : 0u;
#define RFL(x) x = __builtin_amdgcn_readfirstlane(x)
    RFL(mLd); RFL(mSg); RFL(mFl); RFL(mPf); RFL(mEv); RFL(mAs); RFL(mLS);
    asm volatile("" : "+s"(mLd), "+s"(mSg), "+s"(mFl), "+s"(mPf), "+s"(mEv), "+s"(mAs), "+s"(mLS));
#define BIT(m) UNL(((m) >> i) & 1u)
#define RND(off) (i - (off) < pstart ? 0 : ((i - (off) - pstart) >> 3) + 1)
#define Q2 (i + 2 < 32 ? i + 2 : 31)
    const char* pa = cA + 131072;
    const char* pb = cB + 65536;
    for (int i = 0; i < NT / 2 - 1; ++i) {
        SCHED;
        LDB(B0, 0, 0); SCHED; LDA(At, 0, 0); STA(1, 1, pa - 65536 + HTB);
        WAIT_L(8); BAR; WAIT_L(0); MMA(0, 0, At, B0); BAR; SCHED;
        LDB(B1, 0, 1); STB(0, 0, pb); if (BIT(mLd)) LOADW01(RND(0));
        BAR; WAIT_L(0); MMA(0, 1, At, B1); BAR; SCHED;
        LDA(At, 0, 1); STA(0, 0, pa); if (BIT(mLd)) LOADW23(RND(0));
        BAR; WAIT_L(0); MMA(1, 0, At, B0); BAR; SCHED;
        STB(0, 1, pb + HTB);
        if (BIT(mFl)) { FLAGLD(Q2); if (BIT(mLd)) WAIT_V(11); else WAIT_V(7); }
        else if (BIT(mLd)) WAIT_V(10); else WAIT_V(6);
        if (BIT(mSg)) SIGNALB(RND(2));
        if (BIT(mEv)) {
            if (wid < 4) SIGNALA(1u);
            else {
                const unsigned need = (i == 1 && (lane & 31) >= 12) ? 0u : fneed;
                unsigned av; asm volatile("v_mov_b32 %0, v221" : "=v"(av));
                if (!__all((int)(av - need) >= 0)) { if (wait_words((const GAS unsigned*)(myws + aflag64), need, tmo, call)) { if (lane == 0) ctlw[1] = 1u; } WAIT_V(0); }
            }
        }
        BAR; MMA(1, 1, At, B1); BAR; SCHED;
        LDB(B0, 1, 0); SCHED; LDA(At, 1, 0); STA(0, 1, pa + HTB);
        WAIT_L(8); BAR; WAIT_L(0); MMA(0, 0, At, B0); BAR; SCHED;
        LDB(B1, 1, 1); STB(1, 0, pb + 32768);
        if (BIT(mPf)) PF(Q2);
        if (BIT(mLS)) { if (BIT(mLd)) LOADW45(RND(0)); else STOREW01(RND(1)); }
        BAR; WAIT_L(0); MMA(0, 1, At, B1); BAR; SCHED;
        LDA(At, 1, 1); STA(1, 0, pa + 65536);
        if (BIT(mLS)) { if (BIT(mLd)) LOADW67(RND(0)); else STOREW23(RND(1)); }
        BAR; WAIT_L(0); MMA(1, 0, At, B0); BAR; SCHED;
        STB(1, 1, pb + 32768 + HTB);
        pa += 131072; pb += 65536;
        if (BIT(mPf)) { if (BIT(mLS)) WAIT_V(11); else WAIT_V(7); }
        else { if (BIT(mLS)) WAIT_V(10); else WAIT_V(6); if (BIT(mFl)) FLAGCK(Q2); }
        if (BIT(mAs)) asm volatile("global_load_dword v221, %0, %1 sc0 sc1" :: "v"(aflag64), "s"(myws) : "memory");
        BAR; MMA(1, 1, At, B1); BAR; SCHED;
    }
    {
        half8 Aa[2][2], Ab[2][2];
#define LDA1(dst, kt, ai, m) do { _Pragma("unroll") for (int k = 0; k < 2; ++k) dst[kt][k] = *(const LAS half8*)(lds + SA(kt, ai) + aoff + (m) * 2048 + k * 1024); } while (0)
#define MF2(ai, bj, m, n, A, B) asm volatile("v_mfma_f32_16x16x32_f16 %0, %1, %2, %0\n\tv_mfma_f32_16x16x32_f16 %0, %3, %4, %0" : "+v"(acc[ai][bj][m][n]) : "v"(A[0]), "v"(B[n][0]), "v"(A[1]), "v"(B[n][1]))
#define MF2R(ai, bj, m, n, A, R0, R1) asm volatile("v_mfma_f32_16x16x32_f16 %0, %1, " R0 ", %0\n\tv_mfma_f32_16x16x32_f16 %0, %2, " R1 ", %0" : "+v"(acc[ai][bj][m][n]) : "v"(A[0]), "v"(A[1]))
#define MF4(ai, bj, m, n, A, B, R0, R1) asm volatile("v_mfma_f32_16x16x32_f16 %0, %1, %2, %0\n\tv_mfma_f32_16x16x32_f16 %0, %3, %4, %0\n\tv_mfma_f32_16x16x32_f16 %0, %5, " R0 ", %0\n\tv_mfma_f32_16x16x32_f16 %0, %6, " R1 ", %0" \
        : "+v"(acc[ai][bj][m][n]) : "v"(A[0][0]), "v"(B[n][0]), "v"(A[0][1]), "v"(B[n][1]), "v"(A[1][0]), "v"(A[1][1]))
#define MMB(ai, m, A) do { MF4(ai, 0, m, 0, A, B0, "v[224:227]", "v[228:231]"); MF4(ai, 0, m, 1, A, B0, "v[232:235]", "v[236:239]"); \
        MF4(ai, 1, m, 0, A, B1, "v[240:243]", "v[244:247]"); MF4(ai, 1, m, 1, A, B1, "v[248:251]", "v[252:255]"); } while (0)
        LDA1(Aa, 0, 0, 0); LDB(B0, 0, 0); LDB(B1, 0, 1); STA(1, 1, pa - 65536 + HTB);
        if (UNL(bad) && lane == 0) ctlw[0] = 1u;
        if (wr == 0) BAR;
        WAIT_L(0); MF2(0, 0, 0, 0, Aa[0], B0); MF2(0, 0, 0, 1, Aa[0], B0); MF2(0, 1, 0, 0, Aa[0], B1); MF2(0, 1, 0, 1, Aa[0], B1); SCHED;
        WAIT_V(2); BAR; SCHED;
        asm volatile("ds_read_b128 v[224:227], %0\n\tds_read_b128 v[228:231], %0 offset:1024\n\tds_read_b128 v[232:235], %0 offset:2048\n\tds_read_b128 v[236:239], %0 offset:3072\n\t"
                     "ds_read_b128 v[240:243], %0 offset:16384\n\tds_read_b128 v[244:247], %0 offset:17408\n\tds_read_b128 v[248:251], %0 offset:18432\n\tds_read_b128 v[252:255], %0 offset:19456"
                     :: "v"((unsigned)(size_t)lds + SB(1, 0) + boff) : "memory");
        LDA1(Aa, 1, 0, 0);
        bad = *(volatile LAS unsigned*)ctlw | *(volatile LAS unsigned*)(ctlw + 1);
        WAIT_L(0);
        MF2R(0, 0, 0, 0, Aa[1], "v[224:227]", "v[228:231]"); MF2R(0, 0, 0, 1, Aa[1], "v[232:235]", "v[236:239]");
        MF2R(0, 1, 0, 0, Aa[1], "v[240:243]", "v[244:247]"); MF2R(0, 1, 0, 1, Aa[1], "v[248:251]", "v[252:255]"); SCHED;
        LDA1(Aa, 0, 0, 1); LDA1(Aa, 1, 0, 1);
        bad = __builtin_amdgcn_readfirstlane(bad);
        int tz = threadIdx.x; asm volatile("" : "+v"(tz));
        const unsigned ovoff = (unsigned)(((tz >> 4) & 3) * 4 * N_ + (tz & 15) * 4) * 4u;
        const __amdgpu_buffer_rsrc_t ors = __builtin_amdgcn_make_buffer_rsrc(p.out + (size_t)(pm * 256 + wr * 64) * N_ + pn * 256 + wc * 64, 0, 192 * N_ * 4, 0x00020000);
#define EPI(ai, m) do { const float k1 = -2.0f * 0.7978845608028654f * 1.4426950408889634f, k2 = k1 * 0.044715f; f32x4 v[4]; \
            _Pragma("unroll") for (int t = 0; t < 4; ++t) _Pragma("unroll") for (int h = 0; h < 2; ++h) { \
                const f32x2 y = {acc[ai][t >> 1][m][t & 1][2 * h], acc[ai][t >> 1][m][t & 1][2 * h + 1]}; \
                const f32x2 z = y * __builtin_elementwise_fma(y * y, (f32x2){k2, k2}, (f32x2){k1, k1}); \
                f32x2 e = {__builtin_amdgcn_exp2f(z[0]), __builtin_amdgcn_exp2f(z[1])}; e += 1.0f; \
                float o0 = y[0] * __builtin_amdgcn_rcpf(e[0]); float o1 = y[1] * __builtin_amdgcn_rcpf(e[1]); \
                asm("" : "+v"(o0)); asm("" : "+v"(o1)); v[2 * h][t] = o0; v[2 * h + 1][t] = o1; } \
            _Pragma("unroll") for (int j = 0; j < 4; ++j) __builtin_amdgcn_raw_buffer_store_b128(__builtin_bit_cast(u32x4, v[j]), ors, ovoff, ((ai) * 128 + (m) * 16 + j) * N_ * 4, 2); } while (0)
        WAIT_V(0); BAR; SCHED;
        if (UNL(bad)) {
            const unsigned q = 0x7fc00000u;
#pragma unroll 1
            for (int i = 0; i < 32; ++i) __builtin_amdgcn_raw_buffer_store_b128((u32x4){q, q, q, q}, ors, ovoff, ((i >> 4) * 128 + ((i >> 2) & 3) * 16 + (i & 3)) * N_ * 4, 0);
        } else {
            STAMP(3);
            WAIT_L(0); MMB(0, 1, Aa); SCHED;
            EPI(0, 0); SCHED; LDA1(Aa, 0, 0, 2); LDA1(Aa, 1, 0, 2); LDA1(Ab, 0, 0, 3); LDA1(Ab, 1, 0, 3); SCHED;
            WAIT_L(4); MMB(0, 2, Aa); SCHED;
            LDA1(Aa, 0, 1, 0); LDA1(Aa, 1, 1, 0); SCHED; EPI(0, 1); SCHED;
            MMB(0, 3, Ab); SCHED;
            LDA1(Ab, 0, 1, 1); LDA1(Ab, 1, 1, 1); SCHED; EPI(0, 2); SCHED;
            MMB(1, 0, Aa); SCHED;
            LDA1(Aa, 0, 1, 2); LDA1(Aa, 1, 1, 2); SCHED; EPI(0, 3); SCHED;
            MMB(1, 1, Ab); SCHED;
            LDA1(Ab, 0, 1, 3); LDA1(Ab, 1, 1, 3); SCHED; EPI(1, 0); SCHED;
            MMB(1, 2, Aa); SCHED;
            EPI(1, 1); SCHED;
            MMB(1, 3, Ab); SCHED;
            EPI(1, 2); SCHED;
            asm volatile("s_nop 7" ::: "memory"); SCHED;
            EPI(1, 3);
        }
    }
    STAMP(4);
    WAIT_V(0);
    STAMP(5);
}

extern "C" void kernel_launch(int rank, int nranks, ncclComm_t comm, hipStream_t stream,
                              void* const* d_in, const int* in_sizes, int n_in, void* d_out, int out_size,
                              const int* in_dtypes, int out_dtype, void* const* d_out_all, void* const* d_ws_all, size_t ws_bytes) {
    static int ok = -1;
    if (ok < 0) {
        int dev = 0, cus = 0;
        ok = hipGetDevice(&dev) == hipSuccess && hipDeviceGetAttribute(&cus, hipDeviceAttributeMultiprocessorCount, dev) == hipSuccess && cus >= 256 &&
             hipFuncSetAttribute((const void*)ag_gemm, hipFuncAttributeMaxDynamicSharedMemorySize, LDS_BYTES) == hipSuccess;
    }
    if (!ok || nranks != NRANK || (unsigned)rank >= (unsigned)NRANK || ws_bytes < WS_NEED || n_in != 2 || in_sizes[0] != MS * K_ || in_sizes[1] != K_ * N_ || out_size != M_ * N_) {
        fprintf(stderr, "ag_gemm: unsupported configuration\n");
        return;
    }
    Params p; p.x = (const float*)d_in[0]; p.w = (const float*)d_in[1]; p.out = (float*)d_out; p.rank = rank;
    for (int q = 0; q < NRANK; ++q) p.ws[q] = (char*)d_ws_all[q];
    ag_gemm<<<256, 512, LDS_BYTES, stream>>>(p);
}
```

```cpp
#include <hip/hip_runtime.h>
#include <rccl/rccl.h>
#include <cstdio>

#define UNL(x) __builtin_expect(!!(x), 0)
#define LAS __attribute__((address_space(3)))
#define GAS __attribute__((address_space(1)))
typedef _Float16 half8 __attribute__((ext_vector_type(8)));
typedef float f32x4 __attribute__((ext_vector_type(4)));
typedef float f32x2 __attribute__((ext_vector_type(2)));
typedef unsigned u32x4 __attribute__((ext_vector_type(4)));

constexpr int NRANK = 8;
constexpr int M_ = 4096, K_ = 4096, N_ = 4096, MS = M_ / NRANK;
constexpr int NT = K_ / 64;
constexpr int HTB = 16384;
constexpr int LDS_BYTES = 8 * HTB + 512;
constexpr size_t OFF_A16 = 1u << 20;
constexpr size_t A16_BYTES = (size_t)MS * K_ * 2;
constexpr size_t OFF_B16 = 16u << 20;
constexpr size_t WS_NEED = OFF_B16 + (size_t)K_ * N_ * 2;
constexpr unsigned long long DEADLINE = 50000000ull;
constexpr unsigned CB_TMO = 0;
constexpr unsigned CB_TK = 4096;
constexpr unsigned CB_BC = CB_TK + 256 * 128;
constexpr unsigned CB_AC = CB_BC + 512 * 128;
static_assert(CB_AC + 512 * 128 <= OFF_A16 && OFF_A16 + 2 * A16_BYTES <= OFF_B16, "workspace layout");

struct Params { const float* x; const float* w; float* out; char* ws[NRANK]; int rank; };

__device__ __forceinline__ u32x4 cvt8(f32x4 a, f32x4 b) {
    half8 h; h[0] = (_Float16)a[0]; h[1] = (_Float16)a[1]; h[2] = (_Float16)a[2]; h[3] = (_Float16)a[3];
    h[4] = (_Float16)b[0]; h[5] = (_Float16)b[1]; h[6] = (_Float16)b[2]; h[7] = (_Float16)b[3]; return __builtin_bit_cast(u32x4, h);
}
__device__ __forceinline__ bool wait_words(const GAS unsigned* word, unsigned need, unsigned* tmo, unsigned call) {
    const unsigned long long t0 = __builtin_amdgcn_s_memrealtime();
    for (unsigned n = 1;; ++n) {
        const unsigned v = __hip_atomic_load(word, __ATOMIC_RELAXED, __HIP_MEMORY_SCOPE_SYSTEM);
        if (__all((int)(v - need) >= 0)) return false;
        if ((n & 15u) == 0u) {
            if (__builtin_amdgcn_readfirstlane(__hip_atomic_load(tmo, __ATOMIC_RELAXED, __HIP_MEMORY_SCOPE_AGENT)) == call || __builtin_amdgcn_s_memrealtime() - t0 > DEADLINE) {
                __hip_atomic_store(tmo, call, __ATOMIC_RELAXED, __HIP_MEMORY_SCOPE_AGENT); return true; }
        }
        __builtin_amdgcn_s_sleep(1);
    }
}

__global__ __launch_bounds__(512, 2) __attribute__((amdgpu_num_vgpr(110))) void ag_gemm(Params p) {
    extern __shared__ __attribute__((aligned(16))) unsigned char shm[];
    LAS unsigned char* lds = (LAS unsigned char*)shm;
    const int tid = threadIdx.x, wid = __builtin_amdgcn_readfirstlane(tid >> 6), lane = tid & 63;
    const int b = blockIdx.x;
    char* myws = p.ws[p.rank];
    const int wr = wid >> 2, wc = wid & 3;
    const int g = b & 7, idx = b >> 3;
    const int src = (p.rank + g) & (NRANK - 1);
    const int pm = 2 * src + (idx & 1), pn = idx >> 1;
    const int jp = (g + pn) & 7;
    const int mem = jp * 2 + (idx & 1);
    const char* wsrc = (const char*)(p.w + (size_t)(mem * 64 + (wid & 1) * 32) * N_ + pn * 256 + (wid >> 1) * 64);
    const unsigned wvoff = (unsigned)((lane >> 4) * 8 * N_ + (lane & 15) * 4) * 4u;
    char* bdst = myws + OFF_B16 + (size_t)((pn * 64 + mem) * 2) * HTB + ((wid >> 1) * 4 + (wid & 1)) * 1024;
    const unsigned lane16 = lane * 16;
    constexpr size_t WROUND = (size_t)1024 * N_ * 4, BROUND = (size_t)16 * 2 * HTB;
    unsigned* const tmo = (unsigned*)(myws + CB_TMO);
    unsigned long long bctr = (unsigned long long)myws + CB_BC + (pn * 32 + jp) * 128;
    asm volatile("" : "+s"(bctr));
#define WREGS "v224","v225","v226","v227","v228","v229","v230","v231","v232","v233","v234","v235","v236","v237","v238","v239","v240","v241","v242","v243","v244","v245","v246","v247","v248","v249","v250","v251","v252","v253","v254","v255"
#define LW1(R, j) asm volatile("global_load_dwordx4 " R ", %0, %1 nt" :: "v"(wvoff), "s"(wsrc + (size_t)(j) * N_ * 4) : "memory")
#define LOADW0() do { LW1("v[224:227]", 0); LW1("v[228:231]", 1); LW1("v[232:235]", 2); LW1("v[236:239]", 3); LW1("v[240:243]", 4); LW1("v[244:247]", 5); LW1("v[248:251]", 6); LW1("v[252:255]", 7); } while (0)
#define CV1(h, a0, a1, a2, a3, a4, a5, a6, a7) asm volatile("v_cvt_pk_f16_f32 %0, v" #a0 ", v" #a1 "\n\tv_cvt_pk_f16_f32 %1, v" #a2 ", v" #a3 "\n\tv_cvt_pk_f16_f32 %2, v" #a4 ", v" #a5 "\n\tv_cvt_pk_f16_f32 %3, v" #a6 ", v" #a7 \
        : "=v"(h[0]), "=v"(h[1]), "=v"(h[2]), "=v"(h[3]))
#define SCHED __builtin_amdgcn_sched_barrier(0)
#define STOREW01(r) do { const __amdgpu_buffer_rsrc_t _rs = __builtin_amdgcn_make_buffer_rsrc(bdst + (size_t)(r) * BROUND, 0, 4096, 0x00020000); u32x4 h0; u32x4 h1; \
        CV1(h0, 224, 228, 232, 236, 240, 244, 248, 252); CV1(h1, 225, 229, 233, 237, 241, 245, 249, 253); SCHED; \
        __builtin_amdgcn_raw_buffer_store_b128(h0, _rs, lane16, 0, 16); __builtin_amdgcn_raw_buffer_store_b128(h1, _rs, lane16 + 2048, 0, 16); SCHED; asm volatile("s_nop 1"); } while (0)
#define STOREW23(r) do { const __amdgpu_buffer_rsrc_t _rs = __builtin_amdgcn_make_buffer_rsrc(bdst + (size_t)(r) * BROUND + HTB, 0, 4096, 0x00020000); u32x4 h2; u32x4 h3; \
        CV1(h2, 226, 230, 234, 238, 242, 246, 250, 254); CV1(h3, 227, 231, 235, 239, 243, 247, 251, 255); SCHED; \
        __builtin_amdgcn_raw_buffer_store_b128(h2, _rs, lane16, 0, 16); __builtin_amdgcn_raw_buffer_store_b128(h3, _rs, lane16 + 2048, 0, 16); SCHED; asm volatile("s_nop 1"); } while (0)
#define SIGNALB(r) do { if (lane == 0) asm volatile("global_atomic_add %0, %1, %2 sc1" :: "v"(0u), "v"(1u), "s"(bctr + (r) * 1024) : "memory"); } while (0)
#define SIGNALA(n) do { if (lane < NRANK) asm volatile("global_atomic_add v[222:223], %0, off sc1" :: "v"(n) : "memory"); } while (0)
#define WAIT_V(n) asm volatile("s_waitcnt vmcnt(" #n ")" ::: "memory")
#define WAIT_L(n) asm volatile("s_waitcnt lgkmcnt(" #n ")" ::: "memory")
#define BAR __builtin_amdgcn_s_barrier()
    asm volatile("" ::: "v220", "v221", "v222", "v223", WREGS);

    const bool clsA = jp < 2, clsC = jp >= 5;
    const int xb = (7 - jp) * 32 + idx;
    const unsigned ldsw = __builtin_amdgcn_readfirstlane((unsigned)(size_t)lds + (unsigned)wid * 2048u);
    const unsigned pfoff = lane * 64;
    LAS unsigned* ctlw = (LAS unsigned*)(lds + 8 * HTB + 256);
    const char* fB = myws + CB_BC + pn * 32 * 128;
    const unsigned aflag4 = CB_AC + (src * 64 + (lane & 1) * 32 + ((lane >> 1) & 3)) * 128;
    const unsigned aflag64 = CB_AC + (src * 64 + lane) * 128;
    asm volatile("global_load_dword v220, %0, %1 sc1" :: "v"(0u), "s"(myws + CB_TK + b * 128) : "memory");
    unsigned call;
    const char* cA0;
#define CALL_FROM_V220() do { asm volatile("v_readfirstlane_b32 %0, v220" : "=s"(call)); call += 1u; cA0 = p.ws[src] + OFF_A16 + (size_t)(call & 1) * A16_BYTES; } while (0)
#define PFO(off) asm volatile("s_mov_b32 m0, %2\n\ts_nop 0\n\tglobal_load_lds_dword %0, %1" :: "v"(pfoff), "s"(cA0 + (off)), "s"(ldsw - wid * 2048 + 8 * HTB) : "memory")
#define PF(q) PFO(idx * 4096 + (size_t)(q) * 131072)
    if (wid < 4) {
        f32x4 xv[8];
        asm volatile("global_load_dwordx2 v[222:223], %0, %1" :: "v"((lane & 7) * 8), "s"((const char*)__builtin_amdgcn_kernarg_segment_ptr() + __builtin_offsetof(Params, ws)) : "memory");
        if (tid == 0) { ctlw[0] = 0u; ctlw[1] = 0u; ctlw[2] = 0u; WAIT_L(0); }
        if (!clsC) { if (!clsA) __builtin_amdgcn_s_sleep(32); LOADW0(); }
        { const char* s = (const char*)(p.x + (size_t)((xb & 3) * 128 + wid * 32) * K_ + (xb >> 2) * 64);
          const unsigned xo = (unsigned)((lane & 15) * K_ + (lane >> 4) * 8) * 4u;
          asm volatile("global_load_dwordx4 %0, %4, %5\n\tglobal_load_dwordx4 %1, %4, %5 offset:16\n\tglobal_load_dwordx4 %2, %4, %5 offset:128\n\tglobal_load_dwordx4 %3, %4, %5 offset:144"
                       : "=&v"(xv[0]), "=&v"(xv[1]), "=&v"(xv[2]), "=&v"(xv[3]) : "v"(xo), "s"(s) : "memory");
          asm volatile("global_load_dwordx4 %0, %4, %5\n\tglobal_load_dwordx4 %1, %4, %5 offset:16\n\tglobal_load_dwordx4 %2, %4, %5 offset:128\n\tglobal_load_dwordx4 %3, %4, %5 offset:144"
                       : "=&v"(xv[4]), "=&v"(xv[5]), "=&v"(xv[6]), "=&v"(xv[7]) : "v"(xo), "s"(s + (size_t)16 * K_ * 4) : "memory"); }
        BAR;
        if (!clsC) { WAIT_V(8); STOREW01(0); STOREW23(0); WAIT_V(4); } else WAIT_V(0);
        asm volatile("" : "+v"(xv[0]), "+v"(xv[1]), "+v"(xv[2]), "+v"(xv[3]), "+v"(xv[4]), "+v"(xv[5]), "+v"(xv[6]), "+v"(xv[7]) :: "memory");
        CALL_FROM_V220();
        asm volatile("v_lshl_add_u64 v[222:223], v[222:223], 0, %0" :: "s"((unsigned long long)(CB_AC + ((p.rank * 2 + ((xb >> 1) & 1)) * 32 + (xb >> 3)) * 128)));
        { char* dst = myws + OFF_A16 + (size_t)(call & 1) * A16_BYTES + (size_t)xb * HTB + wid * 4096;
          const __amdgpu_buffer_rsrc_t rs = __builtin_amdgcn_make_buffer_rsrc(dst, 0, 4096, 0x00020000);
#pragma unroll
          for (int i = 0; i < 4; ++i) __builtin_amdgcn_raw_buffer_store_b128(cvt8(xv[2 * i], xv[2 * i + 1]), rs, lane16 + i * 1024, 0, 17); }
        SCHED;
        if (jp <= 2) { WAIT_V(4); SIGNALB(0); }
        if (clsC) {
            WAIT_V(0);
            unsigned arr = 0;
            if (lane == 0) arr = __hip_atomic_fetch_add(ctlw + 2, 1u, __ATOMIC_RELAXED, __HIP_MEMORY_SCOPE_WORKGROUP);
            if (__builtin_amdgcn_readfirstlane(arr) == 3u) SIGNALA(4u);
        }
        if (wid == 0) {
            const bool to = wait_words((const GAS unsigned*)(fB + (lane & 1) * 128), 16u * call, tmo, call);
            if (to && lane == 0) ctlw[0] = 1u;
            WAIT_L(0);
        }
    } else {
        if (!clsC) { if (!clsA) __builtin_amdgcn_s_sleep(32); LOADW0(); BAR; WAIT_V(0); STOREW01(0); STOREW23(0); if (jp <= 2) { WAIT_V(0); SIGNALB(0); } }
        else { BAR; WAIT_V(0); }
        CALL_FROM_V220();
        if (wid == 4 || jp == 5) {
            const bool to = wait_words((const GAS unsigned*)(myws + aflag4), 16u * call, tmo, call);
            if (to && lane == 0) ctlw[1] = 1u;
            WAIT_L(0);
            if (jp == 5) {
                const unsigned u = (((pn >> 3) * 2 + (idx & 1)) * 4 + (wid - 4)) * 4096;
                PFO(u); WAIT_V(0); PFO(u + 65536); WAIT_V(0); PFO(u + 131072);
            }
        }
    }
    BAR;
    asm volatile("" ::: "memory");
    if (tid == 0) __hip_atomic_store((unsigned*)(myws + CB_TK + b * 128), call, __ATOMIC_RELAXED, __HIP_MEMORY_SCOPE_AGENT);
    unsigned bad = 0;
    const char* cA = cA0 + (pm & 1) * 2 * HTB;
    const char* cB = myws + OFF_B16 + (size_t)pn * 64 * 2 * HTB;
    const unsigned voff = wid * 2048 + lane * 16;
    const int aoff = wr * 8192 + lane * 16, boff = wc * 4096 + lane * 16;
    const unsigned fneed = 16u * call;
#define SA(b, h) (((b) * 2 + (h)) * HTB)
#define SB(b, h) ((4 + (b) * 2 + (h)) * HTB)
#define STAGE(bufoff, gbase, AUX) asm volatile("s_add_u32 m0, %1, %3\n\ts_nop 0\n\tglobal_load_lds_dwordx4 %0, %2" AUX "\n\tglobal_load_lds_dwordx4 %0, %2 offset:1024" AUX \
        :: "v"(voff), "s"(ldsw), "s"((const char*)(gbase)), "i"(bufoff) : "memory", "scc")
#define STA(b, h, ptr) STAGE(SA(b, h), ptr, "")
#define STB(b, h, ptr) STAGE(SB(b, h), ptr, " sc1")
#define LDA(dst, b, h) do { _Pragma("unroll") for (int m = 0; m < 4; ++m) _Pragma("unroll") for (int k = 0; k < 2; ++k) dst[m][k] = *(const LAS half8*)(lds + SA(b, h) + aoff + m * 2048 + k * 1024); } while (0)
#define LDB(dst, b, h) do { _Pragma("unroll") for (int n = 0; n < 2; ++n) _Pragma("unroll") for (int k = 0; k < 2; ++k) dst[n][k] = *(const LAS half8*)(lds + SB(b, h) + boff + n * 2048 + k * 1024); } while (0)
#define MMA(ai, bj, At, Bt) do { _Pragma("unroll") for (int m = 0; m < 4; ++m) _Pragma("unroll") for (int nn = 0; nn < 2; ++nn) { \
        const int n = (m & 1) ? 1 - nn : nn, k = nn & 1; \
        asm volatile("v_mfma_f32_16x16x32_f16 %0, %1, %2, %0\n\tv_mfma_f32_16x16x32_f16 %0, %3, %4, %0" : "+v"(acc[ai][bj][m][n]) : "v"(At[m][k]), "v"(Bt[n][k]), "v"(At[m][1 - k]), "v"(Bt[n][1 - k])); } } while (0)
#define FLAGLD(q) asm volatile("global_load_dword v220, %0, %1 sc0 sc1" :: "v"(0u), "s"(fB + (q) * 128) : "memory")
#define FLAGCK(q) do { unsigned fv; asm volatile("v_readfirstlane_b32 %0, v220" : "=s"(fv)); if ((int)(fv - fneed) < 0) { if (wait_words((const GAS unsigned*)(fB + (q) * 128), fneed, tmo, call)) bad = 1u; \
        WAIT_V(0); } } while (0)
    f32x4 acc[2][2][4][2];
#pragma unroll
    for (int a = 0; a < 2; ++a)
#pragma unroll
        for (int bb = 0; bb < 2; ++bb)
#pragma unroll
            for (int m = 0; m < 4; ++m)
#pragma unroll
                for (int n = 0; n < 2; ++n) { acc[a][bb][m][n] = (f32x4){0.f, 0.f, 0.f, 0.f}; asm volatile("" : "+v"(acc[a][bb][m][n])); }
    half8 At[4][2], B0[2][2], B1[2][2];
    STB(0, 0, cB); STA(0, 0, cA); STB(0, 1, cB + HTB); STA(0, 1, cA + HTB);
    unsigned mLd = 0x010101u << jp;
    if (clsC) mLd |= 1u;
    unsigned mSt = mLd << 1, mSg = mLd << 2, mLS = mLd | mSt;
    if (jp == 3 || jp == 4) mSg |= 1u;
    unsigned mFl = wr == 0 ? 0x11111111u << wc : 0u;
    unsigned mPf = wr == 1 ? 0x11111111u << wc : 0u;
    unsigned mEv = wid == 4 ? 0x202u : (wid < 4 && !clsC) ? 1u : 0u;
    unsigned mAs = wid == 4 ? 0x101u : 0u;
    unsigned long long wp = (unsigned long long)wsrc + (clsC ? 0 : WROUND), bp = (unsigned long long)bdst + (clsC ? 0 : BROUND), bc = bctr + (jp <= 2 ? 1024 : 0);
#define RFL(x) x = __builtin_amdgcn_readfirstlane(x)
    RFL(mLd); RFL(mSt); RFL(mSg); RFL(mFl); RFL(mPf); RFL(mEv); RFL(mAs); RFL(mLS);
    asm volatile("" : "+s"(mLd), "+s"(mSt), "+s"(mSg), "+s"(mFl), "+s"(mPf), "+s"(mEv), "+s"(mAs), "+s"(mLS), "+s"(wp), "+s"(bp), "+s"(bc));
    if (wr == 1) BAR;
    WAIT_V(4); BAR;
    STB(1, 0, cB + 32768); STA(1, 0, cA + 65536); STB(1, 1, cB + 32768 + HTB);
    WAIT_V(6); BAR;
#define BIT(m) UNL(((m) >> i) & 1u)
#define Q2 (i + 2 < 32 ? i + 2 : 31)
#define LWP(R, j) asm volatile("global_load_dwordx4 " R ", %0, %1 nt" :: "v"(wvoff), "s"(wp + (unsigned long long)(j) * N_ * 4) : "memory")
#define SW1(h, off) asm volatile("global_store_dwordx4 %0, %1, %2 offset:" #off " sc1" :: "v"(lane16), "v"(h), "s"(bq) : "memory")
    const char* pa = cA + 131072;
    const char* pb = cB + 65536;
    for (int i = 0; i < NT / 2 - 1; ++i) {
        SCHED;
        LDB(B0, 0, 0); SCHED; LDA(At, 0, 0); STA(1, 1, pa - 65536 + HTB);
        WAIT_L(8); BAR; WAIT_L(0); MMA(0, 0, At, B0); BAR; SCHED;
        LDB(B1, 0, 1); STB(0, 0, pb);
        if (BIT(mLd)) { LWP("v[224:227]", 0); LWP("v[228:231]", 1); LWP("v[232:235]", 2); LWP("v[236:239]", 3); }
        BAR; WAIT_L(0); MMA(0, 1, At, B1); BAR; SCHED;
        LDA(At, 0, 1); STA(0, 0, pa);
        BAR; WAIT_L(0); MMA(1, 0, At, B0); BAR; SCHED;
        STB(0, 1, pb + HTB);
        if (BIT(mFl)) { FLAGLD(Q2); if (BIT(mLd)) WAIT_V(11); else WAIT_V(7); }
        else if (BIT(mLd)) WAIT_V(10); else WAIT_V(6);
        if (BIT(mSg)) { if (lane == 0) asm volatile("global_atomic_add %0, %1, %2 sc1" :: "v"(0u), "v"(1u), "s"(bc) : "memory"); bc += 1024; }
        if (BIT(mEv)) {
            if (wid < 4) SIGNALA(1u);
            else {
                const unsigned need = (i == 1 && (lane & 31) >= 12) ? 0u : fneed;
                unsigned av; asm volatile("v_mov_b32 %0, v221" : "=v"(av));
                if (!__all((int)(av - need) >= 0)) { if (wait_words((const GAS unsigned*)(myws + aflag64), need, tmo, call)) { if (lane == 0) ctlw[1] = 1u; } WAIT_V(0); }
            }
        }
        BAR; MMA(1, 1, At, B1); BAR; SCHED;
        LDB(B0, 1, 0); SCHED; LDA(At, 1, 0); STA(0, 1, pa + HTB);
        WAIT_L(8); BAR; WAIT_L(0); MMA(0, 0, At, B0); BAR; SCHED;
        LDB(B1, 1, 1); STB(1, 0, pb + 32768);
        if (BIT(mPf)) PF(Q2);
        if (BIT(mLS)) {
            if (BIT(mLd)) { LWP("v[240:243]", 4); LWP("v[244:247]", 5); LWP("v[248:251]", 6); LWP("v[252:255]", 7); }
            else {
                u32x4 h0, h1;
                CV1(h0, 224, 228, 232, 236, 240, 244, 248, 252); CV1(h1, 225, 229, 233, 237, 241, 245, 249, 253);
                { const unsigned long long bq = bp; SW1(h0, 0); SW1(h1, 2048); }
                u32x4 h2, h3;
                asm volatile("s_nop 1");
                CV1(h2, 226, 230, 234, 238, 242, 246, 250, 254); CV1(h3, 227, 231, 235, 239, 243, 247, 251, 255);
                { const unsigned long long bq = bp + HTB; SW1(h2, 0); SW1(h3, 2048); }
                wp += WROUND; bp += BROUND;
                asm volatile("s_nop 1");
            }
        }
        BAR; WAIT_L(0); MMA(0, 1, At, B1); BAR; SCHED;
        LDA(At, 1, 1); STA(1, 0, pa + 65536);
        BAR; WAIT_L(0); MMA(1, 0, At, B0); BAR; SCHED;
        STB(1, 1, pb + 32768 + HTB);
        pa += 131072; pb += 65536;
        if (BIT(mPf)) { if (BIT(mLS)) WAIT_V(11); else WAIT_V(7); }
        else { if (BIT(mLS)) WAIT_V(10); else WAIT_V(6); if (BIT(mFl)) FLAGCK(Q2); }
        if (BIT(mAs)) asm volatile("global_load_dword v221, %0, %1 sc0 sc1" :: "v"(aflag64), "s"(myws) : "memory");
        BAR; MMA(1, 1, At, B1); BAR; SCHED;
    }
    {
        half8 Aa[2][2], Ab[2][2];
#define LDA1(dst, kt, ai, m) do { _Pragma("unroll") for (int k = 0; k < 2; ++k) dst[kt][k] = *(const LAS half8*)(lds + SA(kt, ai) + aoff + (m) * 2048 + k * 1024); } while (0)
#define MF2(ai, bj, m, n, A, B) asm volatile("v_mfma_f32_16x16x32_f16 %0, %1, %2, %0\n\tv_mfma_f32_16x16x32_f16 %0, %3, %4, %0" : "+v"(acc[ai][bj][m][n]) : "v"(A[0]), "v"(B[n][0]), "v"(A[1]), "v"(B[n][1]))
#define MF2R(ai, bj, m, n, A, R0, R1) asm volatile("v_mfma_f32_16x16x32_f16 %0, %1, " R0 ", %0\n\tv_mfma_f32_16x16x32_f16 %0, %2, " R1 ", %0" : "+v"(acc[ai][bj][m][n]) : "v"(A[0]), "v"(A[1]))
#define MF4(ai, bj, m, n, A, B, R0, R1) asm volatile("v_mfma_f32_16x16x32_f16 %0, %1, %2, %0\n\tv_mfma_f32_16x16x32_f16 %0, %3, %4, %0\n\tv_mfma_f32_16x16x32_f16 %0, %5, " R0 ", %0\n\tv_mfma_f32_16x16x32_f16 %0, %6, " R1 ", %0" \
        : "+v"(acc[ai][bj][m][n]) : "v"(A[0][0]), "v"(B[n][0]), "v"(A[0][1]), "v"(B[n][1]), "v"(A[1][0]), "v"(A[1][1]))
#define MMB(ai, m, A) do { MF4(ai, 0, m, 0, A, B0, "v[224:227]", "v[228:231]"); MF4(ai, 0, m, 1, A, B0, "v[232:235]", "v[236:239]"); \
        MF4(ai, 1, m, 0, A, B1, "v[240:243]", "v[244:247]"); MF4(ai, 1, m, 1, A, B1, "v[248:251]", "v[252:255]"); } while (0)
        LDA1(Aa, 0, 0, 0); LDB(B0, 0, 0); LDB(B1, 0, 1); STA(1, 1, pa - 65536 + HTB);
        if (UNL(bad) && lane == 0) ctlw[0] = 1u;
        if (wr == 0) BAR;
        WAIT_L(0); MF2(0, 0, 0, 0, Aa[0], B0); MF2(0, 0, 0, 1, Aa[0], B0); MF2(0, 1, 0, 0, Aa[0], B1); MF2(0, 1, 0, 1, Aa[0], B1); SCHED;
        WAIT_V(2); BAR; SCHED;
        asm volatile("ds_read_b128 v[224:227], %0\n\tds_read_b128 v[228:231], %0 offset:1024\n\tds_read_b128 v[232:235], %0 offset:2048\n\tds_read_b128 v[236:239], %0 offset:3072\n\t"
                     "ds_read_b128 v[240:243], %0 offset:16384\n\tds_read_b128 v[244:247], %0 offset:17408\n\tds_read_b128 v[248:251], %0 offset:18432\n\tds_read_b128 v[252:255], %0 offset:19456"
                     :: "v"((unsigned)(size_t)lds + SB(1, 0) + boff) : "memory");
        LDA1(Aa, 1, 0, 0);
        bad = *(volatile LAS unsigned*)ctlw | *(volatile LAS unsigned*)(ctlw + 1);
        WAIT_L(0);
        MF2R(0, 0, 0, 0, Aa[1], "v[224:227]", "v[228:231]"); MF2R(0, 0, 0, 1, Aa[1], "v[232:235]", "v[236:239]");
        MF2R(0, 1, 0, 0, Aa[1], "v[240:243]", "v[244:247]"); MF2R(0, 1, 0, 1, Aa[1], "v[248:251]", "v[252:255]"); SCHED;
        LDA1(Aa, 0, 0, 1); LDA1(Aa, 1, 0, 1);
        bad = __builtin_amdgcn_readfirstlane(bad);
        int tz = threadIdx.x; asm volatile("" : "+v"(tz));
        const unsigned ovoff = (unsigned)(((tz >> 4) & 3) * 4 * N_ + (tz & 15) * 4) * 4u;
        const __amdgpu_buffer_rsrc_t ors = __builtin_amdgcn_make_buffer_rsrc(p.out + (size_t)(pm * 256 + wr * 64) * N_ + pn * 256 + wc * 64, 0, 192 * N_ * 4, 0x00020000);
#define EPI(ai, m) do { const float k1 = -2.0f * 0.7978845608028654f * 1.4426950408889634f, k2 = k1 * 0.044715f; f32x4 v[4]; \
            _Pragma("unroll") for (int t = 0; t < 4; ++t) _Pragma("unroll") for (int h = 0; h < 2; ++h) { \
                const f32x2 y = {acc[ai][t >> 1][m][t & 1][2 * h], acc[ai][t >> 1][m][t & 1][2 * h + 1]}; \
                const f32x2 z = y * __builtin_elementwise_fma(y * y, (f32x2){k2, k2}, (f32x2){k1, k1}); \
                f32x2 e = {__builtin_amdgcn_exp2f(z[0]), __builtin_amdgcn_exp2f(z[1])}; e += 1.0f; \
                float o0 = y[0] * __builtin_amdgcn_rcpf(e[0]); float o1 = y[1] * __builtin_amdgcn_rcpf(e[1]); \
                asm("" : "+v"(o0)); asm("" : "+v"(o1)); v[2 * h][t] = o0; v[2 * h + 1][t] = o1; } \
            _Pragma("unroll") for (int j = 0; j < 4; ++j) __builtin_amdgcn_raw_buffer_store_b128(__builtin_bit_cast(u32x4, v[j]), ors, ovoff, ((ai) * 128 + (m) * 16 + j) * N_ * 4, 2); } while (0)
        WAIT_V(0); BAR; SCHED;
        if (UNL(bad)) {
            const unsigned q = 0x7fc00000u;
#pragma unroll 1
            for (int i = 0; i < 32; ++i) __builtin_amdgcn_raw_buffer_store_b128((u32x4){q, q, q, q}, ors, ovoff, ((i >> 4) * 128 + ((i >> 2) & 3) * 16 + (i & 3)) * N_ * 4, 0);
        } else {
            WAIT_L(0); MMB(0, 1, Aa); SCHED;
            EPI(0, 0); SCHED; LDA1(Aa, 0, 0, 2); LDA1(Aa, 1, 0, 2); LDA1(Ab, 0, 0, 3); LDA1(Ab, 1, 0, 3); SCHED;
            WAIT_L(4); MMB(0, 2, Aa); SCHED;
            LDA1(Aa, 0, 1, 0); LDA1(Aa, 1, 1, 0); SCHED; EPI(0, 1); SCHED;
            MMB(0, 3, Ab); SCHED;
            LDA1(Ab, 0, 1, 1); LDA1(Ab, 1, 1, 1); SCHED; EPI(0, 2); SCHED;
            MMB(1, 0, Aa); SCHED;
            LDA1(Aa, 0, 1, 2); LDA1(Aa, 1, 1, 2); SCHED; EPI(0, 3); SCHED;
            MMB(1, 1, Ab); SCHED;
            LDA1(Ab, 0, 1, 3); LDA1(Ab, 1, 1, 3); SCHED; EPI(1, 0); SCHED;
            MMB(1, 2, Aa); SCHED;
            EPI(1, 1); SCHED;
            MMB(1, 3, Ab); SCHED;
            EPI(1, 2); SCHED;
            asm volatile("s_nop 7" ::: "memory"); SCHED;
            EPI(1, 3);
        }
    }
    WAIT_V(0);
}

extern "C" void kernel_launch(int rank, int nranks, ncclComm_t comm, hipStream_t stream,
                              void* const* d_in, const int* in_sizes, int n_in, void* d_out, int out_size,
                              const int* in_dtypes, int out_dtype, void* const* d_out_all, void* const* d_ws_all, size_t ws_bytes) {
    static int ok = -1;
    if (ok < 0) {
        int dev = 0, cus = 0;
        ok = hipGetDevice(&dev) == hipSuccess && hipDeviceGetAttribute(&cus, hipDeviceAttributeMultiprocessorCount, dev) == hipSuccess && cus >= 256 &&
             hipFuncSetAttribute((const void*)ag_gemm, hipFuncAttributeMaxDynamicSharedMemorySize, LDS_BYTES) == hipSuccess;
    }
    if (!ok || nranks != NRANK || (unsigned)rank >= (unsigned)NRANK || ws_bytes < WS_NEED || n_in != 2 || in_sizes[0] != MS * K_ || in_sizes[1] != K_ * N_ || out_size != M_ * N_) {
        fprintf(stderr, "ag_gemm: unsupported configuration\n");
        return;
    }
    Params p; p.x = (const float*)d_in[0]; p.w = (const float*)d_in[1]; p.out = (float*)d_out; p.rank = rank;
    for (int q = 0; q < NRANK; ++q) p.ws[q] = (char*)d_ws_all[q];
    ag_gemm<<<256, 512, LDS_BYTES, stream>>>(p);
}
```

```cpp
#include <hip/hip_runtime.h>
#include <rccl/rccl.h>
#include <cstdio>

#define UNL(x) __builtin_expect(!!(x), 0)
#define LAS __attribute__((address_space(3)))
#define GAS __attribute__((address_space(1)))
typedef _Float16 half8 __attribute__((ext_vector_type(8)));
typedef float f32x4 __attribute__((ext_vector_type(4)));
typedef float f32x2 __attribute__((ext_vector_type(2)));
typedef unsigned u32x4 __attribute__((ext_vector_type(4)));

constexpr int NRANK = 8;
constexpr int M_ = 4096, K_ = 4096, N_ = 4096, MS = M_ / NRANK;
constexpr int NT = K_ / 64;
constexpr int HTB = 16384;
constexpr int LDS_BYTES = 8 * HTB + 512;
constexpr size_t OFF_A16 = 1u << 20;
constexpr size_t A16_BYTES = (size_t)MS * K_ * 2;
constexpr size_t OFF_B16 = 16u << 20;
constexpr size_t WS_NEED = OFF_B16 + (size_t)K_ * N_ * 2;
constexpr unsigned long long DEADLINE = 50000000ull;
constexpr unsigned CB_TMO = 0;
constexpr unsigned CB_TK = 4096;
constexpr unsigned CB_BC = CB_TK + 256 * 128;
constexpr unsigned CB_AC = CB_BC + 512 * 128;
static_assert(CB_AC + 512 * 128 <= OFF_A16 && OFF_A16 + 2 * A16_BYTES <= OFF_B16, "workspace layout");

struct Params { const float* x; const float* w; float* out; char* ws[NRANK]; int rank; };

__device__ __forceinline__ u32x4 cvt8(f32x4 a, f32x4 b) {
    half8 h; h[0] = (_Float16)a[0]; h[1] = (_Float16)a[1]; h[2] = (_Float16)a[2]; h[3] = (_Float16)a[3];
    h[4] = (_Float16)b[0]; h[5] = (_Float16)b[1]; h[6] = (_Float16)b[2]; h[7] = (_Float16)b[3]; return __builtin_bit_cast(u32x4, h);
}
__device__ __forceinline__ bool wait_words(const GAS unsigned* word, unsigned need, unsigned* tmo, unsigned call) {
    const unsigned long long t0 = __builtin_amdgcn_s_memrealtime();
    for (unsigned n = 1;; ++n) {
        const unsigned v = __hip_atomic_load(word, __ATOMIC_RELAXED, __HIP_MEMORY_SCOPE_SYSTEM);
        if (__all((int)(v - need) >= 0)) return false;
        if ((n & 15u) == 0u) {
            if (__builtin_amdgcn_readfirstlane(__hip_atomic_load(tmo, __ATOMIC_RELAXED, __HIP_MEMORY_SCOPE_AGENT)) == call || __builtin_amdgcn_s_memrealtime() - t0 > DEADLINE) {
                __hip_atomic_store(tmo, call, __ATOMIC_RELAXED, __HIP_MEMORY_SCOPE_AGENT); return true; }
        }
        __builtin_amdgcn_s_sleep(1);
    }
}

__global__ __launch_bounds__(512, 2) __attribute__((amdgpu_num_vgpr(110))) void ag_gemm(Params p) {
    extern __shared__ __attribute__((aligned(16))) unsigned char shm[];
    LAS unsigned char* lds = (LAS unsigned char*)shm;
    const int tid = threadIdx.x, wid = __builtin_amdgcn_readfirstlane(tid >> 6), lane = tid & 63;
    const int b = blockIdx.x;
    char* myws = p.ws[p.rank];
    const int wr = wid >> 2, wc = wid & 3;
    const int g = b & 7, idx = b >> 3;
    const int src = (p.rank + g) & (NRANK - 1);
    const int pm = 2 * src + (idx & 1), pn = idx >> 1;
    const int jp = (g + pn) & 7;
    const int mem = jp * 2 + (idx & 1);
    const char* wsrc = (const char*)(p.w + (size_t)(mem * 64 + (wid & 1) * 32) * N_ + pn * 256 + (wid >> 1) * 64);
    const unsigned wvoff = (unsigned)((lane >> 4) * 8 * N_ + (lane & 15) * 4) * 4u;
    char* bdst = myws + OFF_B16 + (size_t)((pn * 64 + mem) * 2) * HTB + ((wid >> 1) * 4 + (wid & 1)) * 1024;
    const unsigned lane16 = lane * 16;
    constexpr size_t WROUND = (size_t)1024 * N_ * 4, BROUND = (size_t)16 * 2 * HTB;
    unsigned* const tmo = (unsigned*)(myws + CB_TMO);
    unsigned long long bctr = (unsigned long long)myws + CB_BC + (pn * 32 + jp) * 128;
    asm volatile("" : "+s"(bctr));
#define WREGS "v224","v225","v226","v227","v228","v229","v230","v231","v232","v233","v234","v235","v236","v237","v238","v239","v240","v241","v242","v243","v244","v245","v246","v247","v248","v249","v250","v251","v252","v253","v254","v255"
#define LW1(R, j) asm volatile("global_load_dwordx4 " R ", %0, %1 nt" :: "v"(wvoff), "s"(wsrc + (size_t)(j) * N_ * 4) : "memory")
#define LOADW0() do { LW1("v[224:227]", 0); LW1("v[228:231]", 1); LW1("v[232:235]", 2); LW1("v[236:239]", 3); LW1("v[240:243]", 4); LW1("v[244:247]", 5); LW1("v[248:251]", 6); LW1("v[252:255]", 7); } while (0)
#define CV1(h, a0, a1, a2, a3, a4, a5, a6, a7) asm volatile("v_cvt_pk_f16_f32 %0, v" #a0 ", v" #a1 "\n\tv_cvt_pk_f16_f32 %1, v" #a2 ", v" #a3 "\n\tv_cvt_pk_f16_f32 %2, v" #a4 ", v" #a5 "\n\tv_cvt_pk_f16_f32 %3, v" #a6 ", v" #a7 \
        : "=v"(h[0]), "=v"(h[1]), "=v"(h[2]), "=v"(h[3]))
#define SCHED __builtin_amdgcn_sched_barrier(0)
#define STOREW01(r) do { const __amdgpu_buffer_rsrc_t _rs = __builtin_amdgcn_make_buffer_rsrc(bdst + (size_t)(r) * BROUND, 0, 4096, 0x00020000); u32x4 h0; u32x4 h1; \
        CV1(h0, 224, 228, 232, 236, 240, 244, 248, 252); CV1(h1, 225, 229, 233, 237, 241, 245, 249, 253); SCHED; \
        __builtin_amdgcn_raw_buffer_store_b128(h0, _rs, lane16, 0, 16); __builtin_amdgcn_raw_buffer_store_b128(h1, _rs, lane16 + 2048, 0, 16); SCHED; asm volatile("s_nop 1"); } while (0)
#define STOREW23(r) do { const __amdgpu_buffer_rsrc_t _rs = __builtin_amdgcn_make_buffer_rsrc(bdst + (size_t)(r) * BROUND + HTB, 0, 4096, 0x00020000); u32x4 h2; u32x4 h3; \
        CV1(h2, 226, 230, 234, 238, 242, 246, 250, 254); CV1(h3, 227, 231, 235, 239, 243, 247, 251, 255); SCHED; \
        __builtin_amdgcn_raw_buffer_store_b128(h2, _rs, lane16, 0, 16); __builtin_amdgcn_raw_buffer_store_b128(h3, _rs, lane16 + 2048, 0, 16); SCHED; asm volatile("s_nop 1"); } while (0)
#define SIGNALB(r) do { if (lane == 0) asm volatile("global_atomic_add %0, %1, %2 sc1" :: "v"(0u), "v"(1u), "s"(bctr + (r) * 1024) : "memory"); } while (0)
#define SIGNALA(n) do { if (lane < NRANK) asm volatile("global_atomic_add v[222:223], %0, off sc1" :: "v"(n) : "memory"); } while (0)
#define WAIT_V(n) asm volatile("s_waitcnt vmcnt(" #n ")" ::: "memory")
#define WAIT_L(n) asm volatile("s_waitcnt lgkmcnt(" #n ")" ::: "memory")
#define BAR __builtin_amdgcn_s_barrier()
    asm volatile("" ::: "v220", "v221", "v222", "v223", WREGS);

    const bool clsA = jp < 2, clsC = jp >= 5;
    const int xb = (7 - jp) * 32 + idx;
    const unsigned ldsw = __builtin_amdgcn_readfirstlane((unsigned)(size_t)lds + (unsigned)wid * 2048u);
    const unsigned pfoff = lane * 64;
    LAS unsigned* ctlw = (LAS unsigned*)(lds + 8 * HTB + 256);
    const char* fB = myws + CB_BC + pn * 32 * 128;
    const unsigned aflag4 = CB_AC + (src * 64 + (lane & 1) * 32 + ((lane >> 1) & 3)) * 128;
    const unsigned aflag64 = CB_AC + (src * 64 + lane) * 128;
    asm volatile("global_load_dword v220, %0, %1 sc1" :: "v"(0u), "s"(myws + CB_TK + b * 128) : "memory");
    unsigned call;
    const char* cA0;
#define CALL_FROM_V220() do { asm volatile("v_readfirstlane_b32 %0, v220" : "=s"(call)); call += 1u; cA0 = p.ws[src] + OFF_A16 + (size_t)(call & 1) * A16_BYTES; } while (0)
#define PFO(off) asm volatile("s_mov_b32 m0, %2\n\ts_nop 0\n\tglobal_load_lds_dword %0, %1" :: "v"(pfoff), "s"(cA0 + (off)), "s"(ldsw - wid * 2048 + 8 * HTB) : "memory")
    const unsigned pfoff2 = (lane & 31) * 64;
#define PFK(kt) asm volatile("s_mov_b32 m0, %2\n\ts_nop 0\n\tglobal_load_lds_dword %0, %1" :: "v"(pfoff2), "s"(cA0 + idx * 2048 + (size_t)(kt) * 65536), "s"(ldsw - wid * 2048 + 8 * HTB) : "memory")
    if (wid < 4) {
        f32x4 xv[8];
        asm volatile("global_load_dwordx2 v[222:223], %0, %1" :: "v"((lane & 7) * 8), "s"((const char*)__builtin_amdgcn_kernarg_segment_ptr() + __builtin_offsetof(Params, ws)) : "memory");
        if (tid == 0) { ctlw[0] = 0u; ctlw[1] = 0u; ctlw[2] = 0u; WAIT_L(0); }
        if (!clsC) { if (!clsA) __builtin_amdgcn_s_sleep(32); LOADW0(); }
        { const char* s = (const char*)(p.x + (size_t)((xb & 3) * 128 + wid * 32) * K_ + (xb >> 2) * 64);
          const unsigned xo = (unsigned)((lane & 15) * K_ + (lane >> 4) * 8) * 4u;
          asm volatile("global_load_dwordx4 %0, %4, %5\n\tglobal_load_dwordx4 %1, %4, %5 offset:16\n\tglobal_load_dwordx4 %2, %4, %5 offset:128\n\tglobal_load_dwordx4 %3, %4, %5 offset:144"
                       : "=&v"(xv[0]), "=&v"(xv[1]), "=&v"(xv[2]), "=&v"(xv[3]) : "v"(xo), "s"(s) : "memory");
          asm volatile("global_load_dwordx4 %0, %4, %5\n\tglobal_load_dwordx4 %1, %4, %5 offset:16\n\tglobal_load_dwordx4 %2, %4, %5 offset:128\n\tglobal_load_dwordx4 %3, %4, %5 offset:144"
                       : "=&v"(xv[4]), "=&v"(xv[5]), "=&v"(xv[6]), "=&v"(xv[7]) : "v"(xo), "s"(s + (size_t)16 * K_ * 4) : "memory"); }
        BAR;
        if (!clsC) { WAIT_V(8); STOREW01(0); STOREW23(0); WAIT_V(4); } else WAIT_V(0);
        asm volatile("" : "+v"(xv[0]), "+v"(xv[1]), "+v"(xv[2]), "+v"(xv[3]), "+v"(xv[4]), "+v"(xv[5]), "+v"(xv[6]), "+v"(xv[7]) :: "memory");
        CALL_FROM_V220();
        asm volatile("v_lshl_add_u64 v[222:223], v[222:223], 0, %0" :: "s"((unsigned long long)(CB_AC + ((p.rank * 2 + ((xb >> 1) & 1)) * 32 + (xb >> 3)) * 128)));
        { char* dst = myws + OFF_A16 + (size_t)(call & 1) * A16_BYTES + (size_t)xb * HTB + wid * 4096;
          const __amdgpu_buffer_rsrc_t rs = __builtin_amdgcn_make_buffer_rsrc(dst, 0, 4096, 0x00020000);
#pragma unroll
          for (int i = 0; i < 4; ++i) __builtin_amdgcn_raw_buffer_store_b128(cvt8(xv[2 * i], xv[2 * i + 1]), rs, lane16 + i * 1024, 0, 17); }
        SCHED;
        if (jp <= 2) { WAIT_V(4); SIGNALB(0); }
        if (clsC) {
            WAIT_V(0);
            unsigned arr = 0;
            if (lane == 0) arr = __hip_atomic_fetch_add(ctlw + 2, 1u, __ATOMIC_RELAXED, __HIP_MEMORY_SCOPE_WORKGROUP);
            if (__builtin_amdgcn_readfirstlane(arr) == 3u) SIGNALA(4u);
        }
        if (wid == 0) {
            const bool to = wait_words((const GAS unsigned*)(fB + (lane & 1) * 128), 16u * call, tmo, call);
            if (to && lane == 0) ctlw[0] = 1u;
            WAIT_L(0);
        }
    } else {
        if (!clsC) { if (!clsA) __builtin_amdgcn_s_sleep(32); LOADW0(); BAR; WAIT_V(0); STOREW01(0); STOREW23(0); if (jp <= 2) { WAIT_V(0); SIGNALB(0); } }
        else { BAR; WAIT_V(0); }
        CALL_FROM_V220();
        if (wid == 4 || jp == 5) {
            const bool to = wait_words((const GAS unsigned*)(myws + aflag4), 16u * call, tmo, call);
            if (to && lane == 0) ctlw[1] = 1u;
            WAIT_L(0);
            if (jp == 5) {
                const unsigned u = (((pn >> 3) * 2 + (idx & 1)) * 4 + (wid - 4)) * 4096;
                PFO(u); WAIT_V(0); PFO(u + 65536); WAIT_V(0); PFO(u + 131072);
            }
        }
    }
    BAR;
    asm volatile("" ::: "memory");
    if (tid == 0) __hip_atomic_store((unsigned*)(myws + CB_TK + b * 128), call, __ATOMIC_RELAXED, __HIP_MEMORY_SCOPE_AGENT);
    unsigned bad = 0;
    const char* cA = cA0 + (pm & 1) * 2 * HTB;
    const char* cB = myws + OFF_B16 + (size_t)pn * 64 * 2 * HTB;
    const unsigned voff = wid * 2048 + lane * 16;
    const int aoff = wr * 8192 + lane * 16, boff = wc * 4096 + lane * 16;
    const unsigned fneed = 16u * call;
#define SA(b, h) (((b) * 2 + (h)) * HTB)
#define SB(b, h) ((4 + (b) * 2 + (h)) * HTB)
#define STAGE(bufoff, gbase, AUX) asm volatile("s_add_u32 m0, %1, %3\n\ts_nop 0\n\tglobal_load_lds_dwordx4 %0, %2" AUX "\n\tglobal_load_lds_dwordx4 %0, %2 offset:1024" AUX \
        :: "v"(voff), "s"(ldsw), "s"((const char*)(gbase)), "i"(bufoff) : "memory", "scc")
#define STA(b, h, ptr) STAGE(SA(b, h), ptr, "")
#define STB(b, h, ptr) STAGE(SB(b, h), ptr, " sc1")
#define LDA(dst, b, h) do { _Pragma("unroll") for (int m = 0; m < 4; ++m) _Pragma("unroll") for (int k = 0; k < 2; ++k) dst[m][k] = *(const LAS half8*)(lds + SA(b, h) + aoff + m * 2048 + k * 1024); } while (0)
#define LDB(dst, b, h) do { _Pragma("unroll") for (int n = 0; n < 2; ++n) _Pragma("unroll") for (int k = 0; k < 2; ++k) dst[n][k] = *(const LAS half8*)(lds + SB(b, h) + boff + n * 2048 + k * 1024); } while (0)
#define MMA(ai, bj, At, Bt) do { _Pragma("unroll") for (int m = 0; m < 4; ++m) _Pragma("unroll") for (int nn = 0; nn < 2; ++nn) { \
        const int n = (m & 1) ? 1 - nn : nn, k = nn & 1; \
        asm volatile("v_mfma_f32_16x16x32_f16 %0, %1, %2, %0\n\tv_mfma_f32_16x16x32_f16 %0, %3, %4, %0" : "+v"(acc[ai][bj][m][n]) : "v"(At[m][k]), "v"(Bt[n][k]), "v"(At[m][1 - k]), "v"(Bt[n][1 - k])); } } while (0)
#define FLAGLD(q) asm volatile("global_load_dword v220, %0, %1 sc0 sc1" :: "v"(0u), "s"(fB + (q) * 128) : "memory")
#define FLAGCK(q) do { unsigned fv; asm volatile("v_readfirstlane_b32 %0, v220" : "=s"(fv)); if ((int)(fv - fneed) < 0) { if (wait_words((const GAS unsigned*)(fB + (q) * 128), fneed, tmo, call)) bad = 1u; \
        WAIT_V(0); } } while (0)
    f32x4 acc[2][2][4][2];
#pragma unroll
    for (int a = 0; a < 2; ++a)
#pragma unroll
        for (int bb = 0; bb < 2; ++bb)
#pragma unroll
            for (int m = 0; m < 4; ++m)
#pragma unroll
                for (int n = 0; n < 2; ++n) { acc[a][bb][m][n] = (f32x4){0.f, 0.f, 0.f, 0.f}; asm volatile("" : "+v"(acc[a][bb][m][n])); }
    half8 At[4][2], B0[2][2], B1[2][2];
    STB(0, 0, cB); STA(0, 0, cA); STB(0, 1, cB + HTB); STA(0, 1, cA + HTB);
    unsigned mLd = 0x010101u << (jp + 2);
    if (clsC) mLd |= 1u;
    unsigned mSt = mLd << 1, mSg = mLd << 2, mLS = mLd | mSt;
    if (jp == 3 || jp == 4) mSg |= 1u;
    unsigned mFl = 0x11111111u << (wr == 0 ? wc : (wc + 2) & 3);
    unsigned mPf = wr == 1 ? 0x11111111u << wc : 0u;
    unsigned mEv = wid == 4 ? 0x202u : (wid < 4 && !clsC) ? 1u : 0u;
    unsigned mAs = wid == 4 ? 0x101u : 0u;
    unsigned long long wp = (unsigned long long)wsrc + (clsC ? 0 : WROUND), bp = (unsigned long long)bdst + (clsC ? 0 : BROUND), bc = bctr + (jp <= 2 ? 1024 : 0);
#define RFL(x) x = __builtin_amdgcn_readfirstlane(x)
    RFL(mLd); RFL(mSt); RFL(mSg); RFL(mFl); RFL(mPf); RFL(mEv); RFL(mAs); RFL(mLS);
    asm volatile("" : "+s"(mLd), "+s"(mSt), "+s"(mSg), "+s"(mFl), "+s"(mPf), "+s"(mEv), "+s"(mAs), "+s"(mLS), "+s"(wp), "+s"(bp), "+s"(bc));
    if (wr == 1) BAR;
    WAIT_V(4); BAR;
    STB(1, 0, cB + 32768); STA(1, 0, cA + 65536); STB(1, 1, cB + 32768 + HTB);
    WAIT_V(6); BAR;
#define BIT(m) UNL(((m) >> i) & 1u)
#define Q2 (i + 2 < 32 ? i + 2 : 31)
#define LWP(R, j) asm volatile("global_load_dwordx4 " R ", %0, %1 nt" :: "v"(wvoff), "s"(wp + (unsigned long long)(j) * N_ * 4) : "memory")
#define SW1(h, off) asm volatile("global_store_dwordx4 %0, %1, %2 offset:" #off " sc1" :: "v"(lane16), "v"(h), "s"(bq) : "memory")
    const char* pa = cA + 131072;
    const char* pb = cB + 65536;
    for (int i = 0; i < NT / 2 - 1; ++i) {
        SCHED;
        LDB(B0, 0, 0); SCHED; LDA(At, 0, 0); STA(1, 1, pa - 65536 + HTB);
        WAIT_L(8); BAR; WAIT_L(0); MMA(0, 0, At, B0); BAR; SCHED;
        LDB(B1, 0, 1); STB(0, 0, pb);
        if (wr == 1 && BIT(mFl)) PFK(2 * i + 3);
        if (BIT(mLd)) { LWP("v[224:227]", 0); LWP("v[228:231]", 1); LWP("v[232:235]", 2); LWP("v[236:239]", 3); }
        BAR; WAIT_L(0); MMA(0, 1, At, B1); BAR; SCHED;
        LDA(At, 0, 1); STA(0, 0, pa);
        BAR; WAIT_L(0); MMA(1, 0, At, B0); BAR; SCHED;
        STB(0, 1, pb + HTB);
        if (BIT(mFl)) { if (wr == 0) FLAGLD(Q2); if (BIT(mLd)) WAIT_V(11); else WAIT_V(7); }
        else if (BIT(mLd)) WAIT_V(10); else WAIT_V(6);
        if (BIT(mSg)) { if (lane == 0) asm volatile("global_atomic_add %0, %1, %2 sc1" :: "v"(0u), "v"(1u), "s"(bc) : "memory"); bc += 1024; }
        if (BIT(mEv)) {
            if (wid < 4) SIGNALA(1u);
            else {
                const unsigned need = (i == 1 && (lane & 31) >= 12) ? 0u : fneed;
                unsigned av; asm volatile("v_mov_b32 %0, v221" : "=v"(av));
                if (!__all((int)(av - need) >= 0)) { if (wait_words((const GAS unsigned*)(myws + aflag64), need, tmo, call)) { if (lane == 0) ctlw[1] = 1u; } WAIT_V(0); }
            }
        }
        BAR; MMA(1, 1, At, B1); BAR; SCHED;
        LDB(B0, 1, 0); SCHED; LDA(At, 1, 0); STA(0, 1, pa + HTB);
        WAIT_L(8); BAR; WAIT_L(0); MMA(0, 0, At, B0); BAR; SCHED;
        LDB(B1, 1, 1); STB(1, 0, pb + 32768);
        if (BIT(mPf)) PFK(2 * Q2);
        if (BIT(mLS)) {
            if (BIT(mLd)) { LWP("v[240:243]", 4); LWP("v[244:247]", 5); LWP("v[248:251]", 6); LWP("v[252:255]", 7); }
            else {
                u32x4 h0, h1;
                CV1(h0, 224, 228, 232, 236, 240, 244, 248, 252); CV1(h1, 225, 229, 233, 237, 241, 245, 249, 253);
                { const unsigned long long bq = bp; SW1(h0, 0); SW1(h1, 2048); }
                u32x4 h2, h3;
                asm volatile("s_nop 1");
                CV1(h2, 226, 230, 234, 238, 242, 246, 250, 254); CV1(h3, 227, 231, 235, 239, 243, 247, 251, 255);
                { const unsigned long long bq = bp + HTB; SW1(h2, 0); SW1(h3, 2048); }
                wp += WROUND; bp += BROUND;
                asm volatile("s_nop 1");
            }
        }
        BAR; WAIT_L(0); MMA(0, 1, At, B1); BAR; SCHED;
        LDA(At, 1, 1); STA(1, 0, pa + 65536);
        BAR; WAIT_L(0); MMA(1, 0, At, B0); BAR; SCHED;
        STB(1, 1, pb + 32768 + HTB);
        pa += 131072; pb += 65536;
        if (BIT(mPf)) { if (BIT(mLS)) WAIT_V(11); else WAIT_V(7); }
        else { if (BIT(mLS)) WAIT_V(10); else WAIT_V(6); if (wr == 0 && BIT(mFl)) FLAGCK(Q2); }
        if (BIT(mAs)) asm volatile("global_load_dword v221, %0, %1 sc0 sc1" :: "v"(aflag64), "s"(myws) : "memory");
        BAR; MMA(1, 1, At, B1); BAR; SCHED;
    }
    {
        half8 Aa[2][2], Ab[2][2];
#define LDA1(dst, kt, ai, m) do { _Pragma("unroll") for (int k = 0; k < 2; ++k) dst[kt][k] = *(const LAS half8*)(lds + SA(kt, ai) + aoff + (m) * 2048 + k * 1024); } while (0)
#define MF2(ai, bj, m, n, A, B) asm volatile("v_mfma_f32_16x16x32_f16 %0, %1, %2, %0\n\tv_mfma_f32_16x16x32_f16 %0, %3, %4, %0" : "+v"(acc[ai][bj][m][n]) : "v"(A[0]), "v"(B[n][0]), "v"(A[1]), "v"(B[n][1]))
#define MF2R(ai, bj, m, n, A, R0, R1) asm volatile("v_mfma_f32_16x16x32_f16 %0, %1, " R0 ", %0\n\tv_mfma_f32_16x16x32_f16 %0, %2, " R1 ", %0" : "+v"(acc[ai][bj][m][n]) : "v"(A[0]), "v"(A[1]))
#define MF4(ai, bj, m, n, A, B, R0, R1) asm volatile("v_mfma_f32_16x16x32_f16 %0, %1, %2, %0\n\tv_mfma_f32_16x16x32_f16 %0, %3, %4, %0\n\tv_mfma_f32_16x16x32_f16 %0, %5, " R0 ", %0\n\tv_mfma_f32_16x16x32_f16 %0, %6, " R1 ", %0" \
        : "+v"(acc[ai][bj][m][n]) : "v"(A[0][0]), "v"(B[n][0]), "v"(A[0][1]), "v"(B[n][1]), "v"(A[1][0]), "v"(A[1][1]))
#define MMB(ai, m, A) do { MF4(ai, 0, m, 0, A, B0, "v[224:227]", "v[228:231]"); MF4(ai, 0, m, 1, A, B0, "v[232:235]", "v[236:239]"); \
        MF4(ai, 1, m, 0, A, B1, "v[240:243]", "v[244:247]"); MF4(ai, 1, m, 1, A, B1, "v[248:251]", "v[252:255]"); } while (0)
        LDA1(Aa, 0, 0, 0); LDB(B0, 0, 0); LDB(B1, 0, 1); STA(1, 1, pa - 65536 + HTB);
        if (UNL(bad) && lane == 0) ctlw[0] = 1u;
        if (wr == 0) BAR;
        WAIT_L(0); MF2(0, 0, 0, 0, Aa[0], B0); MF2(0, 0, 0, 1, Aa[0], B0); MF2(0, 1, 0, 0, Aa[0], B1); MF2(0, 1, 0, 1, Aa[0], B1); SCHED;
        WAIT_V(2); BAR; SCHED;
        asm volatile("ds_read_b128 v[224:227], %0\n\tds_read_b128 v[228:231], %0 offset:1024\n\tds_read_b128 v[232:235], %0 offset:2048\n\tds_read_b128 v[236:239], %0 offset:3072\n\t"
                     "ds_read_b128 v[240:243], %0 offset:16384\n\tds_read_b128 v[244:247], %0 offset:17408\n\tds_read_b128 v[248:251], %0 offset:18432\n\tds_read_b128 v[252:255], %0 offset:19456"
                     :: "v"((unsigned)(size_t)lds + SB(1, 0) + boff) : "memory");
        LDA1(Aa, 1, 0, 0);
        bad = *(volatile LAS unsigned*)ctlw | *(volatile LAS unsigned*)(ctlw + 1);
        WAIT_L(0);
        MF2R(0, 0, 0, 0, Aa[1], "v[224:227]", "v[228:231]"); MF2R(0, 0, 0, 1, Aa[1], "v[232:235]", "v[236:239]");
        MF2R(0, 1, 0, 0, Aa[1], "v[240:243]", "v[244:247]"); MF2R(0, 1, 0, 1, Aa[1], "v[248:251]", "v[252:255]"); SCHED;
        LDA1(Aa, 0, 0, 1); LDA1(Aa, 1, 0, 1);
        bad = __builtin_amdgcn_readfirstlane(bad);
        int tz = threadIdx.x; asm volatile("" : "+v"(tz));
        const unsigned ovoff = (unsigned)(((tz >> 4) & 3) * 4 * N_ + (tz & 15) * 4) * 4u;
        const __amdgpu_buffer_rsrc_t ors = __builtin_amdgcn_make_buffer_rsrc(p.out + (size_t)(pm * 256 + wr * 64) * N_ + pn * 256 + wc * 64, 0, 192 * N_ * 4, 0x00020000);
#define EPI(ai, m) do { const float k1 = -2.0f * 0.7978845608028654f * 1.4426950408889634f, k2 = k1 * 0.044715f; f32x4 v[4]; \
            _Pragma("unroll") for (int t = 0; t < 4; ++t) _Pragma("unroll") for (int h = 0; h < 2; ++h) { \
                const f32x2 y = {acc[ai][t >> 1][m][t & 1][2 * h], acc[ai][t >> 1][m][t & 1][2 * h + 1]}; \
                const f32x2 z = y * __builtin_elementwise_fma(y * y, (f32x2){k2, k2}, (f32x2){k1, k1}); \
                f32x2 e = {__builtin_amdgcn_exp2f(z[0]), __builtin_amdgcn_exp2f(z[1])}; e += 1.0f; \
                float o0 = y[0] * __builtin_amdgcn_rcpf(e[0]); float o1 = y[1] * __builtin_amdgcn_rcpf(e[1]); \
                asm("" : "+v"(o0)); asm("" : "+v"(o1)); v[2 * h][t] = o0; v[2 * h + 1][t] = o1; } \
            _Pragma("unroll") for (int j = 0; j < 4; ++j) __builtin_amdgcn_raw_buffer_store_b128(__builtin_bit_cast(u32x4, v[j]), ors, ovoff, ((ai) * 128 + (m) * 16 + j) * N_ * 4, 2); } while (0)
        WAIT_V(0); BAR; SCHED;
        if (UNL(bad)) {
            const unsigned q = 0x7fc00000u;
#pragma unroll 1
            for (int i = 0; i < 32; ++i) __builtin_amdgcn_raw_buffer_store_b128((u32x4){q, q, q, q}, ors, ovoff, ((i >> 4) * 128 + ((i >> 2) & 3) * 16 + (i & 3)) * N_ * 4, 0);
        } else {
            WAIT_L(0); MMB(0, 1, Aa); SCHED;
            EPI(0, 0); SCHED; LDA1(Aa, 0, 0, 2); LDA1(Aa, 1, 0, 2); LDA1(Ab, 0, 0, 3); LDA1(Ab, 1, 0, 3); SCHED;
            WAIT_L(4); MMB(0, 2, Aa); SCHED;
            LDA1(Aa, 0, 1, 0); LDA1(Aa, 1, 1, 0); SCHED; EPI(0, 1); SCHED;
            MMB(0, 3, Ab); SCHED;
            LDA1(Ab, 0, 1, 1); LDA1(Ab, 1, 1, 1); SCHED; EPI(0, 2); SCHED;
            MMB(1, 0, Aa); SCHED;
            LDA1(Aa, 0, 1, 2); LDA1(Aa, 1, 1, 2); SCHED; EPI(0, 3); SCHED;
            MMB(1, 1, Ab); SCHED;
            LDA1(Ab, 0, 1, 3); LDA1(Ab, 1, 1, 3); SCHED; EPI(1, 0); SCHED;
            MMB(1, 2, Aa); SCHED;
            EPI(1, 1); SCHED;
            MMB(1, 3, Ab); SCHED;
            EPI(1, 2); SCHED;
            asm volatile("s_nop 7" ::: "memory"); SCHED;
            EPI(1, 3);
        }
    }
    WAIT_V(0);
}

extern "C" void kernel_launch(int rank, int nranks, ncclComm_t comm, hipStream_t stream,
                              void* const* d_in, const int* in_sizes, int n_in, void* d_out, int out_size,
                              const int* in_dtypes, int out_dtype, void* const* d_out_all, void* const* d_ws_all, size_t ws_bytes) {
    static int ok = -1;
    if (ok < 0) {
        int dev = 0, cus = 0;
        ok = hipGetDevice(&dev) == hipSuccess && hipDeviceGetAttribute(&cus, hipDeviceAttributeMultiprocessorCount, dev) == hipSuccess && cus >= 256 &&
             hipFuncSetAttribute((const void*)ag_gemm, hipFuncAttributeMaxDynamicSharedMemorySize, LDS_BYTES) == hipSuccess;
    }
    if (!ok || nranks != NRANK || (unsigned)rank >= (unsigned)NRANK || ws_bytes < WS_NEED || n_in != 2 || in_sizes[0] != MS * K_ || in_sizes[1] != K_ * N_ || out_size != M_ * N_) {
        fprintf(stderr, "ag_gemm: unsupported configuration\n");
        return;
    }
    Params p; p.x = (const float*)d_in[0]; p.w = (const float*)d_in[1]; p.out = (float*)d_out; p.rank = rank;
    for (int q = 0; q < NRANK; ++q) p.ws[q] = (char*)d_ws_all[q];
    ag_gemm<<<256, 512, LDS_BYTES, stream>>>(p);
}
```
